```python
import numpy as np

kernel_name = "gpu_a2a_gemm_m4096_k4096_n4096_e4m3e4m3_dequant"


ABSMAX_RTOL = 0.03

COMPUTE_DTYPE = "float32"
EPILOGUE = "dequant"
IN_QUANT = None
OUT_QUANT = None
IN_FP8 = "e4m3e4m3"
IN_DTYPES = {"x": "e4m3", "w_mat": "e4m3",
             "scale_x": "float32", "scale_w": "float32"}
WORLD_SIZE = 8

def _snap_fp8(x, mant_bits, max_abs, min_normal):
    a = np.abs(x.astype(np.float64))
    _m, _e = np.frexp(a)
    q = np.where(a >= min_normal, np.ldexp(1.0, _e - 1 - mant_bits),
                 min_normal * 2.0 ** -mant_bits)
    snapped = np.minimum(np.round(a / q) * q, max_abs)
    return (np.sign(x) * snapped).astype(np.float32)

def setup_inputs():

    x = _snap_fp8(np.random.randn(4096, 4096).astype(np.float32), 3, 448.0, 2.0 ** -6)
    w_mat = _snap_fp8(np.random.randn(4096, 4096).astype(np.float32), 3, 448.0, 2.0 ** -6)
    scale_x = (0.02 * (1.0 + np.random.rand(1))).astype(np.float32)
    scale_w = (0.02 * (1.0 + np.random.rand(1))).astype(np.float32)
    return {"x": x, "w_mat": w_mat, "scale_x": scale_x, "scale_w": scale_w}

def reference(x, w_mat, scale_x, scale_w):
    acc = (x @ w_mat).astype(np.float32)
    y = (acc * (scale_x * scale_w)).astype(np.float32)
    return y
```

```cpp
#include <hip/hip_runtime.h>
#include <rccl/rccl.h>
#include <cstdio>
#include <cstdint>

#define HIP_CHECK(expr) do { hipError_t _e = (expr); if (_e != hipSuccess) { \
    fprintf(stderr, "HIP error %s at %s:%d\n", hipGetErrorString(_e), __FILE__, __LINE__); \
    return; } } while (0)

typedef int      v2i __attribute__((ext_vector_type(2)));
typedef int      v4i __attribute__((ext_vector_type(4)));
typedef unsigned v4u __attribute__((ext_vector_type(4)));
typedef int      v8i __attribute__((ext_vector_type(8)));
typedef float    v4f __attribute__((ext_vector_type(4)));
#define LAS __attribute__((address_space(3)))
#define GAS __attribute__((address_space(1)))

constexpr int NR = 8;
constexpr int GN = 4096;
constexpr int KS = 512;
constexpr int ML = 512;
constexpr int NWG = 256, NTHREADS = 512;
constexpr int NBLK = 8;

constexpr unsigned WS_CNT  = 0;
constexpr unsigned WS_TMO  = 8192;
constexpr unsigned WS_FLAG = 16384;
constexpr unsigned WS_X    = 65536;
constexpr unsigned X_SRC   = ML * KS;
constexpr unsigned X_PAR   = NR * X_SRC;
constexpr unsigned WS_END  = WS_X + 2 * X_PAR;

constexpr unsigned L_WLOC = 0;
constexpr unsigned L_PRIV = 32768;
constexpr unsigned L_SYNC = L_PRIV + 7 * 16384;
constexpr unsigned L_END  = L_SYNC + 256;
constexpr unsigned SY_LOC = 0, SY_PART = 4, SY_RED = 4 + 4 * NBLK;

constexpr unsigned long long DEADLINE_TICKS = 20000000ull;

#define PUSH_WIN 3
#define OUT_AUX 16

struct Params {
    const unsigned char* x;
    const unsigned char* w;
    const float* sx;
    const float* sw;
    float* out;
    unsigned char* ws[NR];
    int rank;
    int pad;
};

#define WAIT_V(n) asm volatile("s_waitcnt vmcnt(%0)" :: "n"(n) : "memory")
#define WAIT_L0() asm volatile("s_waitcnt lgkmcnt(0)" ::: "memory")

#define STAMP(k) do { } while (0)
#define DIAG_DUMP() do { } while (0)

__device__ __forceinline__ void glds16(const unsigned char* sbase, unsigned voff, unsigned lds_addr) {
    asm volatile("s_mov_b32 m0, %2\n\ts_nop 0\n\tglobal_load_lds_dwordx4 %0, %1"
                 :: "v"(voff), "s"(sbase), "s"(lds_addr) : "memory");
}
__device__ __forceinline__ void stage_w(const unsigned char* src, unsigned ve, unsigned vo, unsigned lds_addr) {
#pragma unroll
    for (int n = 0; n < 8; ++n) glds16(src + (size_t)n * 16 * GN, (n & 1) ? vo : ve, lds_addr + n * 1024);
}
template <int OFF>
__device__ __forceinline__ void gload16(v4i& dst, const unsigned char* sbase, unsigned voff) {
    asm volatile("global_load_dwordx4 %0, %1, %2 offset:%3" : "=v"(dst) : "v"(voff), "s"(sbase), "n"(OFF) : "memory");
}
__device__ __forceinline__ void push16(unsigned char* sbase, unsigned voff, const v4i& v) {
    asm volatile("global_store_dwordx4 %0, %1, %2 sc0 sc1\n\ts_nop 1" :: "v"(voff), "v"(v), "s"(sbase) : "memory");
}
__device__ __forceinline__ void flag_store(unsigned char* sbase, unsigned zero, unsigned val) {
    asm volatile("s_mov_b64 exec, 1\n\tglobal_store_dword %0, %1, %2 sc0 sc1\n\ts_mov_b64 exec, -1"
                 :: "v"(zero), "v"(val), "s"(sbase) : "memory");
}
__device__ __forceinline__ v8i cat8(const v4i& a, const v4i& b) {
    return __builtin_shufflevector(a, b, 0, 1, 2, 3, 4, 5, 6, 7);
}
__device__ __forceinline__ v2i tr8(unsigned addr) {
    return __builtin_amdgcn_ds_read_tr8_b64_v2i32((LAS v2i*)addr);
}
__device__ __forceinline__ v8i wfrag(unsigned a) {
    v2i r0 = tr8(a), r1 = tr8(a + 512), r2 = tr8(a + 4096), r3 = tr8(a + 4608);
    v4i lo = __builtin_shufflevector(r0, r1, 0, 1, 2, 3), hi = __builtin_shufflevector(r2, r3, 0, 1, 2, 3);
    return cat8(lo, hi);
}
__device__ __forceinline__ v4f mfma(const v8i& a, const v8i& b, const v4f& c) {
    return __builtin_amdgcn_mfma_scale_f32_16x16x128_f8f6f4(a, b, c, 0, 0, 0, 0, 0, 0);
}
__device__ __forceinline__ void lds_signal(unsigned addr) {
    unsigned one = 1;
    asm volatile("s_waitcnt lgkmcnt(0)\n\ts_mov_b64 exec, 1\n\tds_add_u32 %0, %1\n\ts_mov_b64 exec, -1"
                 :: "v"(addr), "v"(one) : "memory");
}
__device__ __forceinline__ void lds_wait(unsigned addr, unsigned target) {
    for (;;) {
        unsigned v;
        asm volatile("ds_read_b32 %0, %1\n\ts_waitcnt lgkmcnt(0)" : "=v"(v) : "v"(addr) : "memory");
        if ((unsigned)__builtin_amdgcn_readfirstlane(v) >= target) break;
        __builtin_amdgcn_s_sleep(1);
    }
}

constexpr int push_younger(int j) {
    int n = 0;
    if (j < PUSH_WIN) {
        n += PUSH_WIN - 1 - j;
        for (int i = 0; i < j; ++i) n += 1 + (i + PUSH_WIN < NBLK ? 1 : 0);
    } else {
        for (int i = j - PUSH_WIN + 1; i < j; ++i) n += 1 + (i + PUSH_WIN < NBLK ? 1 : 0);
    }
    return n;
}
template <int J>
__device__ __forceinline__ void push_tail(unsigned char* dst, unsigned char* flag, unsigned lane16, unsigned zero,
                                          unsigned base, const v4i (&d)[NBLK]
                                          ) {
    if constexpr (J < NBLK) {
        WAIT_V(push_younger(J));
        flag_store(flag, zero, base + J + 1);
        if constexpr (J + PUSH_WIN < NBLK) push16(dst + (J + PUSH_WIN) * 32768, lane16, d[J + PUSH_WIN]);
        STAMP(4 + J);
        push_tail<J + 1>(dst, flag, lane16, zero, base, d
                         );
    }
}

__global__ void __launch_bounds__(NTHREADS, 2) a2a_gemm(Params p) {
    extern __shared__ __attribute__((aligned(16))) unsigned char smem[];
    const unsigned lds0 = (unsigned)(uintptr_t)(LAS unsigned char*)smem;
    const int lane = threadIdx.x & 63;
    const int wid = __builtin_amdgcn_readfirstlane(threadIdx.x >> 6);
    const int b = blockIdx.x, xg = b & 7, bi = b >> 3;
    const int rank = p.rank;
    unsigned char* const myws = p.ws[rank];
    STAMP(0);

    const unsigned zero = 0;
    unsigned char* const cnt_ptr = myws + WS_CNT + (b * 8 + wid) * 4;
    unsigned cnt_v;
    asm volatile("global_load_dword %0, %1, %2 sc1" : "=v"(cnt_v) : "v"(zero), "s"(cnt_ptr) : "memory");
    const unsigned xoff = (lane & 15) * KS + (lane >> 4) * 16;
    const unsigned lane16 = lane * 16;

    if (wid == 0) {
        __builtin_amdgcn_s_setprio(3);
        const int d = (xg + bi) & 7;
        const unsigned peer = (unsigned)(rank + d) & 7u;
        const int pt = bi >> 3, pu = bi & 7;
        v4i dat[NBLK];
        const unsigned char* src = p.x + (size_t)(peer * ML + 16 * pt) * KS + 64 * pu;
#pragma unroll
        for (int j = 0; j < NBLK; ++j) gload16<0>(dat[j], src + j * 64 * KS, xoff);
        __builtin_amdgcn_s_barrier();
        asm volatile("s_waitcnt vmcnt(8)" : "+v"(cnt_v) :: "memory");
        const unsigned call = __builtin_amdgcn_readfirstlane(cnt_v) + 1;
        STAMP(1);
        if (d != 0 && peer < (unsigned)NR) {
            unsigned char* pws = p.ws[peer];
            unsigned char* dst = pws + WS_X + (call & 1) * X_PAR + rank * X_SRC + pt * 8192 + pu * 1024;
            unsigned char* flag = pws + WS_FLAG + (rank * 4 + pt) * 128 + pu * 4;
#pragma unroll
            for (int j = 0; j < PUSH_WIN; ++j) {
                asm volatile("s_waitcnt vmcnt(7)" : "+v"(dat[j]) :: "memory");
                push16(dst + j * 32768, lane16, dat[j]);
            }
            STAMP(2);
            asm volatile("s_waitcnt vmcnt(%8)" : "+v"(dat[0]), "+v"(dat[1]), "+v"(dat[2]), "+v"(dat[3]),
                         "+v"(dat[4]), "+v"(dat[5]), "+v"(dat[6]), "+v"(dat[7]) : "n"(PUSH_WIN) : "memory");
            STAMP(3);
            push_tail<0>(dst, flag, lane16, zero, (call - 1) * NBLK, dat
                         );
        }
        WAIT_V(0);
        STAMP(12);
        flag_store(cnt_ptr, zero, call);
        DIAG_DUMP();
        __builtin_amdgcn_endpgm();
    }
    {
        const int t = bi & 3, nt = xg * 8 + (bi >> 2);
        const unsigned src_rank = (unsigned)(rank + wid) & 7u;
        const unsigned priv = lds0 + L_PRIV + (wid - 1) * 16384;
        const unsigned sync = lds0 + L_SYNC;
        const unsigned ve = (lane >> 2) * GN + (((lane & 3) ^ ((lane >> 4) & 1)) * 16);
        const unsigned vo = ve ^ 32;
        const unsigned char* wcol = p.w + nt * 64;
        const unsigned char* xloc = p.x + (size_t)(rank * ML + 16 * t) * KS;

        if (wid == 1) *(LAS unsigned*)(sync + lane * 4) = 0;
        if (wid <= 4) stage_w(wcol + (size_t)(rank * KS + (wid - 1) * 128) * GN, ve, vo, lds0 + L_WLOC + (wid - 1) * 8192);
        const unsigned char* wown = wcol + (size_t)(src_rank * KS) * GN;
        stage_w(wown, ve, vo, priv);
        stage_w(wown + (size_t)128 * GN, ve, vo, priv + 8192);
        v4i xl[8], x0[2];
        {
            const unsigned char* xs = xloc + (size_t)wid * 64 * KS;
            gload16<0>(xl[0], xs, xoff);   gload16<64>(xl[1], xs, xoff);
            gload16<128>(xl[2], xs, xoff); gload16<192>(xl[3], xs, xoff);
            gload16<256>(xl[4], xs, xoff); gload16<320>(xl[5], xs, xoff);
            gload16<384>(xl[6], xs, xoff); gload16<448>(xl[7], xs, xoff);
            const unsigned char* xz = xloc + ((wid - 1) & 3) * 128;
            gload16<0>(x0[0], xz, xoff);   gload16<64>(x0[1], xz, xoff);
        }
        WAIT_L0();
        __builtin_amdgcn_s_barrier();
        STAMP(1);

        const int g = lane >> 4, q = (lane & 15) >> 1, pp = lane & 1;
        const int f = ((q >> 2) & 1) | ((g & 1) << 1);
        unsigned ta[4];
#pragma unroll
        for (int c = 0; c < 4; ++c) ta[c] = (16 * g + q) * 64 + 8 * pp + ((c ^ f) * 16);

        v8i wf[4][4];
        WAIT_V(18);
        STAMP(2);
        if (wid <= 4) lds_signal(sync + SY_LOC);
#pragma unroll
        for (int c = 0; c < 4; ++c) wf[0][c] = wfrag(priv + ta[c]);
        asm volatile("s_waitcnt lgkmcnt(0)" : "+v"(wf[0][0]), "+v"(wf[0][1]), "+v"(wf[0][2]), "+v"(wf[0][3]) :: "memory");
        stage_w(wown + (size_t)256 * GN, ve, vo, priv);
        WAIT_V(18);
#pragma unroll
        for (int c = 0; c < 4; ++c) wf[1][c] = wfrag(priv + 8192 + ta[c]);
        asm volatile("s_waitcnt lgkmcnt(0)" : "+v"(wf[1][0]), "+v"(wf[1][1]), "+v"(wf[1][2]), "+v"(wf[1][3]) :: "memory");
        stage_w(wown + (size_t)384 * GN, ve, vo, priv + 8192);
        STAMP(3);
        asm volatile("s_waitcnt vmcnt(16)"
                     : "+v"(xl[0]), "+v"(xl[1]), "+v"(xl[2]), "+v"(xl[3]), "+v"(xl[4]), "+v"(xl[5]), "+v"(xl[6]), "+v"(xl[7]),
                       "+v"(x0[0]), "+v"(x0[1]), "+v"(cnt_v) :: "memory");
        const unsigned call = __builtin_amdgcn_readfirstlane(cnt_v) + 1;
        lds_wait(sync + SY_LOC, 4);
        STAMP(4);
        v4f init[4], acc[4];
#pragma unroll
        for (int c = 0; c < 4; ++c) { init[c] = (v4f){0, 0, 0, 0}; acc[c] = (v4f){0, 0, 0, 0}; }
#pragma unroll
        for (int s = 0; s < 4; ++s) {
            v8i wl[4];
#pragma unroll
            for (int c = 0; c < 4; ++c) wl[c] = wfrag(lds0 + L_WLOC + s * 8192 + ta[c]);
#pragma unroll
            for (int c = 0; c < 4; ++c) init[c] = mfma(wl[c], cat8(xl[2 * s], xl[2 * s + 1]), init[c]);
            if (wid == s + 1) {
#pragma unroll
                for (int c = 0; c < 4; ++c) acc[c] = mfma(wl[c], cat8(x0[0], x0[1]), acc[c]);
            }
        }
        STAMP(5);
        WAIT_V(8);
#pragma unroll
        for (int c = 0; c < 4; ++c) wf[2][c] = wfrag(priv + ta[c]);
        WAIT_V(0);
#pragma unroll
        for (int c = 0; c < 4; ++c) wf[3][c] = wfrag(priv + 8192 + ta[c]);
        STAMP(6);

        const __amdgpu_buffer_rsrc_t xr = __builtin_amdgcn_make_buffer_rsrc(
            myws + WS_X + (call & 1) * X_PAR + src_rank * X_SRC + t * 8192, 0, X_SRC - t * 8192, 0x00020000);
        const __amdgpu_buffer_rsrc_t orr = __builtin_amdgcn_make_buffer_rsrc(
            p.out + (size_t)(16 * t) * GN + nt * 64, 0, (ML - 16 * t) * GN * 4 - nt * 256, 0x00020000);
        GAS unsigned* poll_ptr = (GAS unsigned*)(myws + (lane < 8 ? WS_FLAG + (src_rank * 4 + t) * 128 + lane * 4 : WS_TMO));
        const unsigned fbase = (call - 1) * NBLK;
        const float scale = p.sx[0] * p.sw[0];
        const int m = lane & 15;
        const unsigned pw = priv + g * 256 + ((m ^ g) * 16);
        const int rm = 4 * (wid - 1) + g;
        const unsigned rd = lds0 + L_PRIV + m * 256 + ((rm ^ m) * 16);
        const unsigned ooff = rm * GN * 4 + m * 16;
        bool dead = false;
        unsigned prog = 0;

        auto poll = [&]() {
            unsigned v = __hip_atomic_load(poll_ptr, __ATOMIC_RELAXED, __HIP_MEMORY_SCOPE_SYSTEM);
            if (__builtin_amdgcn_readlane(v, 8) == call) dead = true;
            unsigned mn = __builtin_amdgcn_readlane(v, 0);
#pragma unroll
            for (int u = 1; u < 8; ++u) mn = min(mn, (unsigned)__builtin_amdgcn_readlane(v, u));
            prog = mn > fbase ? mn - fbase : 0;
            asm volatile("" ::: "memory");
        };
        auto wait_blk = [&](unsigned j) {
            if (prog > j || dead) return;
            const unsigned long long t0 = __builtin_amdgcn_s_memrealtime();
            for (;;) {
                poll();
                if (prog > j || dead) break;
                if (__builtin_amdgcn_s_memrealtime() - t0 > DEADLINE_TICKS) {
                    __hip_atomic_store((GAS unsigned*)(myws + WS_TMO), call, __ATOMIC_RELAXED, __HIP_MEMORY_SCOPE_AGENT);
                    dead = true;
                    break;
                }
                __builtin_amdgcn_s_sleep(1);
            }
        };
        auto load_x = [&](v8i (&xx)[4], int j) {
            if (!dead) {
#pragma unroll
                for (int s = 0; s < 4; ++s) {
                    v4i lo = __builtin_bit_cast(v4i, __builtin_amdgcn_raw_buffer_load_b128(
                        xr, lane16 + (s & 1) * 2048, j * 32768 + (s >> 1) * 4096, 17));
                    v4i hi = __builtin_bit_cast(v4i, __builtin_amdgcn_raw_buffer_load_b128(
                        xr, lane16 + (s & 1) * 2048 + 1024, j * 32768 + (s >> 1) * 4096, 17));
                    xx[s] = cat8(lo, hi);
                }
            } else {
#pragma unroll
                for (int s = 0; s < 4; ++s) xx[s] = (v8i){-1, -1, -1, -1, -1, -1, -1, -1};
            }
        };
        auto block = [&](v8i (&cur)[4], v8i (&nxt)[4], int j) {
            bool pre = false;
            if (j + 1 < NBLK) {
                if (prog <= (unsigned)j + 1 && !dead) poll();
                if (prog > (unsigned)j + 1 || dead) {
                    load_x(nxt, j + 1);
                    pre = true;
                    asm volatile("" :: "v"(cur[0]), "v"(cur[1]), "v"(cur[2]), "v"(cur[3]));
                }
            }
#pragma unroll
            for (int s = 0; s < 4; ++s)
#pragma unroll
                for (int c = 0; c < 4; ++c) acc[c] = mfma(wf[s][c], cur[s], acc[c]);
            if (j >= 4) lds_wait(sync + SY_RED + (j - 4) * 4, 4);
            const unsigned slot = (j & 3) * 4096;
#pragma unroll
            for (int c = 0; c < 4; ++c)
                *(LAS v4f*)(((pw + slot) ^ (c * 64)) + c * 1024) = acc[c];
            lds_signal(sync + SY_PART + j * 4);
            v4f res;
            if (wid <= 4) {
                lds_wait(sync + SY_PART + j * 4, 7);
                v4f part[7];
#pragma unroll
                for (int w = 0; w < 7; ++w) part[w] = *(LAS v4f*)(rd + slot + w * 16384);
                res = ((part[0] + part[1]) + (part[2] + part[3])) + ((part[4] + part[5]) + part[6]);
                res *= scale;
                asm volatile("" : "+v"(res));
                lds_signal(sync + SY_RED + j * 4);
            }
            if (j + 1 < NBLK && !pre) {
                wait_blk(j + 1);
                load_x(nxt, j + 1);
            }
            if (wid <= 4)
                __builtin_amdgcn_raw_buffer_store_b128(__builtin_bit_cast(v4u, res), orr, ooff, j * 64 * GN * 4, OUT_AUX);
            const bool mine = (j + 1 == wid);
#pragma unroll
            for (int c = 0; c < 4; ++c) {
                acc[c] = mine ? init[c] : (v4f){0, 0, 0, 0};
                asm volatile("" : "+v"(acc[c]));
            }
        };

        v8i xa[4], xb[4];
        wait_blk(0);
        STAMP(7);
        load_x(xa, 0);
#pragma unroll 1
        for (int j = 0; j < NBLK; j += 2) {
            block(xa, xb, j);
            STAMP(8 + j);
            block(xb, xa, j + 1);
            STAMP(9 + j);
        }
        flag_store(cnt_ptr, zero, call);
    }
    DIAG_DUMP();
}

extern "C" void kernel_launch(int rank, int nranks, ncclComm_t comm,
                              hipStream_t stream,
                              void* const* d_in, const int* in_sizes, int n_in,
                              void* d_out, int out_size,
                              const int* in_dtypes, int out_dtype,
                              void* const* d_out_all,
                              void* const* d_ws_all, size_t ws_bytes) {
    (void)comm; (void)in_dtypes; (void)out_dtype; (void)d_out_all;
    if (nranks != NR || (unsigned)rank >= (unsigned)NR || n_in != 4 || ws_bytes < WS_END ||
        in_sizes[0] != 4096 * KS || in_sizes[1] != 4096 * GN || out_size != ML * GN) {
        fprintf(stderr, "unexpected problem shape\n");
        return;
    }
    HIP_CHECK(hipFuncSetAttribute((const void*)a2a_gemm, hipFuncAttributeMaxDynamicSharedMemorySize, L_END));
    Params p;
    p.x = (const unsigned char*)d_in[0];
    p.w = (const unsigned char*)d_in[1];
    p.sx = (const float*)d_in[2];
    p.sw = (const float*)d_in[3];
    p.out = (float*)d_out;
    for (int i = 0; i < NR; ++i) p.ws[i] = (unsigned char*)d_ws_all[i];
    p.rank = rank;
    p.pad = 0;
    a2a_gemm<<<NWG, NTHREADS, L_END, stream>>>(p);
    HIP_CHECK(hipGetLastError());
}
```

```cpp
#include <hip/hip_runtime.h>
#include <rccl/rccl.h>
#include <cstdio>
#include <cstdint>

#pragma clang diagnostic ignored "-Winline-asm"
#pragma clang diagnostic ignored "-Wint-to-pointer-cast"

#define HIP_CHECK(expr) do { hipError_t _e = (expr); if (_e != hipSuccess) { \
    fprintf(stderr, "HIP error %s at %s:%d\n", hipGetErrorString(_e), __FILE__, __LINE__); \
    return; } } while (0)

#define LAS __attribute__((address_space(3)))
#define GAS __attribute__((address_space(1)))

constexpr int NR = 8;
constexpr int GN = 4096;
constexpr int KS = 512;
constexpr int ML = 512;
constexpr int NWG = 256, NTHREADS = 512, NPUSH = 32;

#define ROT_S 12
#define SLOT_T0 40
#define SLOT_DT 70
#define A_LAG 1400
#define A_DELAY 0

constexpr unsigned WS_TMO = 8192;
constexpr unsigned WS_XF  = 16384;
constexpr unsigned WS_CNT = 32768;
constexpr unsigned WS_PF  = 65536;
constexpr unsigned WS_X   = 262144;
constexpr unsigned X_SRC  = ML * KS;
constexpr unsigned X_PAR  = NR * X_SRC;
constexpr unsigned WS_P   = WS_X + 2 * X_PAR;
constexpr unsigned WS_END = WS_P + ML * GN * 4;

constexpr unsigned L_END = 8 * 16384;
static_assert(ROT_S >= 9 && ROT_S <= 20, "the last three blocks of every CU must be blocks it owns");

constexpr unsigned long long DEADLINE_TICKS = 20000000ull;

constexpr int R_POLL = 20, R_SC = 21, R_X = 32, R_W = 128;

struct Params {
    const unsigned char* x;
    const unsigned char* w;
    const float* sx;
    const float* sw;
    float* out;
    unsigned char* myws;
    unsigned char* ws[NR];
    int rank;
    int pad;
    unsigned sched[NPUSH];
};

#define WAIT_V(n) asm volatile("s_waitcnt vmcnt(%0)" :: "n"(n) : "memory")
#define WAIT_L0() asm volatile("s_waitcnt lgkmcnt(0)" ::: "memory")
#define RT32() ((unsigned)__builtin_amdgcn_s_memrealtime())

#define STAMP(k) do { } while (0)
#define DIAG_DUMP() do { } while (0)

__device__ __forceinline__ void glds16(const unsigned char* sbase, unsigned voff, unsigned lds_addr) {
    asm volatile("s_mov_b32 m0, %2\n\ts_nop 0\n\tglobal_load_lds_dwordx4 %0, %1"
                 :: "v"(voff), "s"(sbase), "s"(lds_addr) : "memory");
}
__device__ __forceinline__ void flag_store(unsigned char* sbase, unsigned val) {
    const unsigned zero = 0;
    asm volatile("s_mov_b64 exec, 1\n\tglobal_store_dword %0, %1, %2 sc0 sc1\n\ts_mov_b64 exec, -1"
                 :: "v"(zero), "v"(val), "s"(sbase) : "memory");
}
template <int R, int OFF>
__device__ __forceinline__ void st_sys(unsigned char* sbase, unsigned voff) {
    asm volatile("global_store_dwordx4 %0, v[%c2:%c3], %1 offset:%c4 sc0 sc1\n\ts_nop 1"
                 :: "v"(voff), "s"(sbase), "n"(R), "n"(R + 3), "n"(OFF) : "memory");
}
template <int R, int OFF>
__device__ __forceinline__ void lds_rd128(unsigned addr) {
    asm volatile("ds_read_b128 v[%c1:%c2], %0 offset:%c3" :: "v"(addr), "n"(R), "n"(R + 3), "n"(OFF) : "memory");
}
template <int R, int OFF>
__device__ __forceinline__ void lds_tr8(unsigned addr) {
    asm volatile("ds_read_b64_tr_b8 v[%c1:%c2], %0 offset:%c3" :: "v"(addr), "n"(R), "n"(R + 1), "n"(OFF) : "memory");
}
template <int Q, int C>
__device__ __forceinline__ void w_read(unsigned tb) {
    if constexpr (C < 8) {
        const unsigned a = tb ^ (C * 16);
        lds_tr8<R_W + ((Q >> 1) * 8 + C) * 8 + (Q & 1) * 4, (Q & 1) * 8192>(a);
        lds_tr8<R_W + ((Q >> 1) * 8 + C) * 8 + (Q & 1) * 4 + 2, (Q & 1) * 8192 + 1024>(a);
        w_read<Q, C + 1>(tb);
    }
}
__device__ __forceinline__ void w_stage(const unsigned char* src, unsigned v0, unsigned v1, unsigned lds_addr) {
#pragma unroll
    for (int n = 0; n < 8; ++n) glds16(src + (size_t)n * 8 * GN, (n & 2) ? v1 : v0, lds_addr + n * 1024);
}

#define ASM_STAMP ".macro STAMP idx\n.endm\n"
#define ABL 0
#define STR2(x) #x
#define STR(x) STR2(x)
#define STREAM_ASM \
    ".set ABL, " STR(ABL) "\n" \
    ASM_STAMP \
    ".macro MM0 c, x\n v_mfma_f32_16x16x128_f8f6f4 v[96+4*\\c:99+4*\\c], v[128+8*\\c:135+8*\\c], v[\\x:\\x+7], 0\n.endm\n" \
    ".macro MM1 c, x\n .if (ABL & 8)\n v_mfma_f32_16x16x128_f8f6f4 v[96+4*((\\c+4)&7):99+4*((\\c+4)&7)], v[192+8*\\c:199+8*\\c], v[\\x+8:\\x+15], v[96+4*((\\c+4)&7):99+4*((\\c+4)&7)]\n .else\n v_mfma_f32_16x16x128_f8f6f4 v[96+4*\\c:99+4*\\c], v[192+8*\\c:199+8*\\c], v[\\x+8:\\x+15], v[96+4*\\c:99+4*\\c]\n .endif\n.endm\n" \
    ".macro WR c, slot\n" \
    " .if (ABL & 1)\n .elseif (\\c & 1)\n ds_write_b128 %[wa1], v[96+4*\\c:99+4*\\c] offset:\\slot+(\\c>>1)*128\n" \
    " .else\n ds_write_b128 %[wa0], v[96+4*\\c:99+4*\\c] offset:\\slot+(\\c>>1)*128\n .endif\n.endm\n" \
    ".macro RD w, slot\n" \
    " .if (ABL & 2)\n .elseif (\\w < 4)\n ds_read_b128 v[64+4*\\w:67+4*\\w], %[ra0] offset:\\slot+\\w*16384\n" \
    " .else\n ds_read_b128 v[64+4*\\w:67+4*\\w], %[ra1] offset:\\slot+(\\w-4)*16384\n .endif\n.endm\n" \
    ".macro ADD4 d, a, b\n .if (ABL & 4)\n .elseif (ABL & 64)\n v_add_f32 v[\\d], v[\\a], v[\\b]\n v_add_f32 v[\\d+1], v[\\a+1], v[\\b+1]\n v_add_f32 v[\\d+2], v[\\a+2], v[\\b+2]\n v_add_f32 v[\\d+3], v[\\a+3], v[\\b+3]\n .else\n v_pk_add_f32 v[\\d:\\d+1], v[\\a:\\a+1], v[\\b:\\b+1]\n v_pk_add_f32 v[\\d+2:\\d+3], v[\\a+2:\\a+3], v[\\b+2:\\b+3]\n .endif\n.endm\n" \
    ".macro LX x, u\n .if (ABL & 32)\n global_load_dword v22, %[vzero], %[dummy]\n .else\n global_load_dwordx4 v[\\x+4*\\u:\\x+4*\\u+3], v23, %[xbase] offset:1024*\\u sc0 sc1\n .endif\n.endm\n" \
    ".macro POLL\n global_load_dword v20, %[pa], %[myws] sc0 sc1\n.endm\n" \
    ".macro NEED\n"                                     \
    " s_cbranch_scc1 1f\n s_call_b64 %[ret], .La2a_spin\n1:\n.endm\n" \
    ".macro WORD\n"                                     \
    " v_mov_b32 v23, %[t1]\n s_mov_b64 exec, 1\n global_store_dword v23, %[vcall], %[myws] sc0 sc1\n s_mov_b64 exec, -1\n.endm\n" \
    ".macro FILL\n global_load_dword v22, %[vzero], %[dummy]\n.endm\n" \
    ".macro FINISH p\n ADD4 24, 24, \\p\n v_mul_f32 v24, v21, v24\n v_mul_f32 v25, v21, v25\n v_mul_f32 v26, v21, v26\n v_mul_f32 v27, v21, v27\n.endm\n" \
    ".macro ITER xc, xn, sc, sp, lc, lp\n" \
    " s_waitcnt vmcnt(3)\n" \
    " s_add_u32 %[t0], %[i], 4\n STAMP %[t0]\n" \
    " MM0 0, \\xc\n" \
    " v_cmp_le_u32 vcc, %[call], v20\n" \
    " WR 5, \\sp\n" \
    " s_add_u32 %[t0], %[i], 1\n" \
    " MM1 0, \\xc\n" \
    " s_or_b32 %[t1], vcc_lo, %[lastbit]\n s_bitcmp1_b32 %[t1], %[t0]\n NEED\n" \
    " s_cmp_lt_u32 %[t0], %[nblk]\n s_cselect_b32 %[t1], %[rbn], %[rb0]\n s_lshl_b32 %[t1], %[t1], 12\n" \
    " v_add_u32 v23, %[t1], %[l16]\n" \
    " WR 6, \\sp\n" \
    " LX \\xn, 0\n" \
    " MM0 1, \\xc\n" \
    " WR 7, \\sp\n" \
    " LX \\xn, 1\n" \
    " MM1 1, \\xc\n" \
    " LX \\xn, 2\n" \
    " s_lshl_b32 %[t1], %[rb0], 7\n s_add_u32 %[t1], %[t1], %[pf0]\n" \
    " MM0 2, \\xc\n" \
    " LX \\xn, 3\n" \
    " v_writelane_b32 %[pa], %[t1], \\lc\n" \
    " MM1 2, \\xc\n" \
    " POLL\n" \
    " s_waitcnt vmcnt(5) lgkmcnt(0)\n .if !(ABL & 16)\n s_barrier\n .endif\n" \
    " MM0 3, \\xc\n" \
    " RD 0, \\sp\n RD 1, \\sp\n WR 0, \\sc\n" \
    " s_sub_u32 %[t0], %[i], 3\n s_lshl_b32 %[t1], %[rb3], 7\n s_add_u32 %[t1], %[t1], %[pf0]\n" \
    " s_cmp_lt_u32 %[t0], %[npub]\n" \
    " MM1 3, \\xc\n" \
    " RD 2, \\sp\n RD 3, \\sp\n" \
    " s_cbranch_scc0 6f\n WORD\n s_branch 7f\n" \
    "6:\n FILL\n" \
    "7:\n" \
    " MM0 4, \\xc\n" \
    " WR 1, \\sc\n" \
    " s_cmp_lt_u32 %[i], 2\n s_cbranch_scc0 2f\n" \
    " s_cmp_eq_u32 %[i], %[icnt]\n s_cbranch_scc1 8f\n FILL\n s_branch 4f\n" \
    "8:\n s_mov_b32 %[t1], %[cnt]\n WORD\n s_branch 4f\n"        \
    "2:\n" \
    " s_sub_u32 %[t0], %[i], 2\n s_lshl_b32 %[t1], %[rb2], 18\n v_add_u32 v23, %[t1], %[ooff]\n" \
    " s_cmp_lt_u32 %[t0], %[np]\n s_cselect_b64 %[sel], %[pbase], %[obase]\n s_cbranch_scc1 3f\n" \
    " FINISH 28\n" \
    "3:\n" \
    " global_store_dwordx4 v23, v[24:27], %[sel] sc0 sc1\n" \
    "4:\n" \
    " MM1 4, \\xc\n" \
    " RD 4, \\sp\n RD 5, \\sp\n" \
    " s_cmp_le_u32 %[i], %[np]\n s_cbranch_scc0 5f\n FILL\n s_branch 9f\n" \
    "5:\n" \
    " s_movk_i32 %[t0], \\lp\n s_bitcmp1_b32 vcc_lo, \\lp\n NEED\n" \
    " s_lshl_b32 %[t1], %[rb1], 18\n v_add_u32 v23, %[t1], %[ooff]\n" \
    " global_load_dwordx4 v[28:31], v23, %[pload] sc0 sc1\n" \
    "9:\n" \
    " MM0 5, \\xc\n" \
    " RD 6, \\sp\n RD 7, \\sp\n WR 2, \\sc\n" \
    " MM1 5, \\xc\n" \
    " s_mov_b32 %[rb3], %[rb2]\n s_mov_b32 %[rb2], %[rb1]\n s_mov_b32 %[rb1], %[rb0]\n s_mov_b32 %[rb0], %[rbn]\n" \
    " s_waitcnt lgkmcnt(6)\n" \
    " ADD4 64, 64, 68\n ADD4 72, 72, 76\n" \
    " MM0 6, \\xc\n" \
    " WR 3, \\sc\n" \
    " s_add_u32 %[rbn], %[rbn], %[stride]\n s_and_b32 %[rbn], %[rbn], 31\n s_add_u32 %[i], %[i], 1\n" \
    " ADD4 64, 64, 72\n" \
    " MM1 6, \\xc\n" \
    " s_waitcnt lgkmcnt(2)\n" \
    " ADD4 80, 80, 84\n ADD4 88, 88, 92\n" \
    " MM0 7, \\xc\n" \
    " WR 4, \\sc\n" \
    " ADD4 80, 80, 88\n" \
    " MM1 7, \\xc\n" \
    " ADD4 24, 64, 80\n" \
    " s_cmp_lt_u32 %[i], %[nblk]\n" \
    ".endm\n" \
    ".macro DRAIN sp, lp\n"                             \
    " WR 5, \\sp\n" \
    " s_waitcnt vmcnt(3)\n" \
    " v_cmp_le_u32 vcc, %[call], v20\n" \
    " s_movk_i32 %[t0], \\lp\n s_lshl_b32 %[t1], %[rb1], 18\n v_add_u32 %[l16], %[t1], %[ooff]\n" \
    " s_bitcmp1_b32 vcc_lo, \\lp\n NEED\n" \
    " global_load_dwordx4 v[32:35], %[l16], %[pload] sc0 sc1\n" \
    " s_nop 7\n" \
    " WR 6, \\sp\n WR 7, \\sp\n" \
    " s_lshl_b32 %[t1], %[rb2], 18\n v_add_u32 v23, %[t1], %[ooff]\n" \
    " s_waitcnt vmcnt(1)\n" \
    " FINISH 28\n" \
    " global_store_dwordx4 v23, v[24:27], %[obase] sc0 sc1\n" \
    " s_waitcnt lgkmcnt(0)\n s_barrier\n" \
    " RD 0, \\sp\n RD 1, \\sp\n RD 2, \\sp\n RD 3, \\sp\n RD 4, \\sp\n RD 5, \\sp\n RD 6, \\sp\n RD 7, \\sp\n" \
    " s_waitcnt lgkmcnt(4)\n" \
    " ADD4 64, 64, 68\n ADD4 72, 72, 76\n ADD4 64, 64, 72\n" \
    " s_waitcnt lgkmcnt(0)\n" \
    " ADD4 80, 80, 84\n ADD4 88, 88, 92\n ADD4 80, 80, 88\n ADD4 24, 64, 80\n" \
    " s_waitcnt vmcnt(1)\n" \
    " FINISH 32\n" \
    " global_store_dwordx4 %[l16], v[24:27], %[obase] sc0 sc1\n" \
    " s_branch .La2a_done\n" \
    ".endm\n" \
      \
    " v_mul_f32 v21, v21, v22\n" \
    " s_mov_b32 %[i], 0\n s_mov_b32 %[rb1], 0\n s_mov_b32 %[rb2], 0\n s_mov_b32 %[rb3], 0\n" \
    " s_add_u32 %[rbn], %[rb0], %[stride]\n s_and_b32 %[rbn], %[rbn], 31\n" \
    " s_mov_b32 %[t0], 0\n s_call_b64 %[ret], .La2a_spin\n" \
    " s_lshl_b32 %[t1], %[rb0], 12\n v_add_u32 v23, %[t1], %[l16]\n" \
    " LX 32, 0\n LX 32, 1\n LX 32, 2\n LX 32, 3\n POLL\n" \
    " s_waitcnt vmcnt(0) lgkmcnt(0)\n" \
    ".La2a_loop:\n" \
    " ITER 32, 48, 0, 8192, 16, 18\n" \
    " s_cbranch_scc0 .La2a_drain0\n" \
    " ITER 48, 32, 8192, 0, 18, 16\n" \
    " s_cbranch_scc1 .La2a_loop\n" \
    " DRAIN 8192, 18\n" \
    ".La2a_drain0:\n" \
    " DRAIN 0, 16\n" \
      \
    ".La2a_spin:\n" \
    " s_memrealtime %[tm0]\n" \
    ".La2a_spin1:\n" \
    " POLL\n" \
    " s_waitcnt vmcnt(0)\n" \
    " v_cmp_le_u32 vcc, %[call], v20\n" \
    " s_bitcmp1_b32 vcc_lo, %[t0]\n s_cbranch_scc1 .La2a_spin3\n" \
    " s_bitcmp1_b32 vcc_lo, 17\n s_cbranch_scc1 .La2a_spin2\n" \
    " s_memrealtime %[tm1]\n s_waitcnt lgkmcnt(0)\n" \
    " s_mov_b64 vcc, %[tm1]\n s_mov_b32 %[t1], vcc_lo\n s_mov_b64 vcc, %[tm0]\n s_sub_u32 %[t1], %[t1], vcc_lo\n" \
    " s_cmp_lt_u32 %[t1], %[ddl]\n s_sleep 1\n s_cbranch_scc1 .La2a_spin1\n" \
    " s_movk_i32 %[t1], %[tmo]\n WORD\n s_waitcnt vmcnt(0)\n" \
    ".La2a_spin2:\n" \
    " s_mov_b64 %[xbase], %[dummy]\n s_mov_b64 %[pload], %[dummy]\n s_mov_b64 vcc, 0\n" \
    " v_mov_b32 v21, -1\n v_mov_b32 v128, -1\n v_mov_b32 v136, -1\n v_mov_b32 v144, -1\n v_mov_b32 v152, -1\n" \
    " v_mov_b32 v160, -1\n v_mov_b32 v168, -1\n v_mov_b32 v176, -1\n v_mov_b32 v184, -1\n" \
    ".La2a_spin3:\n" \
    " s_waitcnt lgkmcnt(0)\n" \
    " s_setpc_b64 %[ret]\n" \
    ".La2a_done:\n" \
    " s_nop 1\n"

__global__ void __launch_bounds__(NTHREADS, 2) __attribute__((amdgpu_num_vgpr(10))) a2a_gemm(Params p) {
    extern __shared__ __attribute__((aligned(16))) unsigned char smem[];
    asm volatile("" ::: "v255");
    const unsigned tstart = RT32();
    const unsigned lds0 = (unsigned)(uintptr_t)(LAS unsigned char*)smem;
    const int lane = threadIdx.x & 63;
    const int wid = __builtin_amdgcn_readfirstlane(threadIdx.x >> 6);
    const int b = blockIdx.x;
    const int rank = p.rank;
    unsigned char* const myws = p.myws;
    const unsigned priv = lds0 + wid * 16384;
    const unsigned lane16 = lane * 16;
    const unsigned peer = (unsigned)(rank + wid) & 7u;
    unsigned char* const cnt_ptr = myws + WS_CNT + b * 128;
    {
        const unsigned zero = 0;
        asm volatile("global_load_dword v%c2, %0, %1 sc1" :: "v"(zero), "s"(cnt_ptr), "n"(R_POLL) : "memory");
    }

    if (b < NPUSH) {
        const unsigned sc = p.sched[b];
        unsigned char* const pws = p.ws[wid];
        const unsigned vsrc = (lane >> 4) * KS + (((lane & 15) ^ (lane >> 4)) * 16);
        const unsigned rd = priv + (lane & 15) * 256 + (((lane >> 4) ^ (lane & 15)) * 16);
#pragma unroll
        for (int e = 0; e < 2; ++e) {
            const unsigned u = (sc >> (16 * e)) & 0xffffu, h = u & 1, rb = (u >> 1) & 31;
            const unsigned char* src = p.x + (size_t)(peer * ML + rb * 16) * KS + h * 256;
#pragma unroll
            for (int n = 0; n < 4; ++n) glds16(src + n * 4 * KS, vsrc ^ (n * 64), priv + e * 4096 + n * 1024);
        }
        WAIT_V(4);
        unsigned call;
        asm volatile("v_readfirstlane_b32 %0, v%c1" : "=s"(call) : "n"(R_POLL) : "memory");
        call += 1;
        STAMP(1);
#pragma unroll 1
        for (int e = 0; e < 2; ++e) {
            const unsigned u = (sc >> (16 * e)) & 0xffffu, h = u & 1, rb = (u >> 1) & 31, cls = (u >> 6) & 7, idx = (u >> 9) & 15;
            const unsigned a = rd + e * 4096;
            lds_rd128<R_X, 0>(a); lds_rd128<R_X + 4, 0>(a ^ 64); lds_rd128<R_X + 8, 0>(a ^ 128); lds_rd128<R_X + 12, 0>(a ^ 192);
            unsigned char* dst = pws + WS_X + (call & 1) * X_PAR + rank * X_SRC + (h * 32 + rb) * 4096;
            const unsigned tslot = tstart + SLOT_T0 + (unsigned)(b + e * NPUSH) * SLOT_DT / 10;
            while ((int)(RT32() - tslot) < 0) __builtin_amdgcn_s_sleep(1);
            WAIT_L0();
            st_sys<R_X, 0>(dst, lane16); st_sys<R_X + 4, 1024>(dst, lane16);
            st_sys<R_X + 8, 2048>(dst, lane16); st_sys<R_X + 12, 3072>(dst, lane16);
            STAMP(2 + 2 * e);
            WAIT_V(0);
            flag_store(pws + WS_XF + (rank * 8 + cls) * 128 + idx * 4, call);
            STAMP(3 + 2 * e);
            if (e == 0) {
                __builtin_amdgcn_s_barrier();
                if (wid == 0) flag_store(cnt_ptr, call);
            }
        }
        DIAG_DUMP();
        __builtin_amdgcn_endpgm();
    }

    const int ci = (b >> 3) - NPUSH / 8, cls = ci >> 2, nt = (b & 7) * 4 + (ci & 3);
    const bool isB = cls < 3;
    const int sub = isB ? cls : cls - 3, stride = isB ? 3 : 4, rot = isB ? ROT_S : 0;
    const int nblk = isB ? (cls < 2 ? 11 : 10) : 8;
    const int g = lane >> 4, m = lane & 15;

    {
        const unsigned zero = 0;
        asm volatile("global_load_dword v%c3, %0, %1\n\tglobal_load_dword v%c4, %0, %2"
                     :: "v"(zero), "s"(p.sx), "s"(p.sw), "n"(R_SC), "n"(R_SC + 1) : "memory");
    }
    if (A_DELAY > 0 && !isB) { while ((int)(RT32() - tstart) < A_DELAY) __builtin_amdgcn_s_sleep(1); }
    {
        const unsigned char* wsrc = p.w + (size_t)(peer * KS + (isB ? 256 : 0)) * GN + nt * 128;
        const unsigned v0 = (lane >> 3) * GN + (((lane & 7) ^ g) * 16), v1 = v0 ^ 64;
        const int q = m >> 1;
        const unsigned tb = priv + (16 * g + q) * 128 + 8 * (lane & 1) + ((((q >> 1) & 3) | ((g & 1) << 2)) * 16);
        w_stage(wsrc, v0, v1, priv);
        w_stage(wsrc + (size_t)64 * GN, v0, v1, priv + 8192);
        STAMP(1);
        WAIT_V(8);
        w_read<0, 0>(tb);
        WAIT_L0();
        w_stage(wsrc + (size_t)128 * GN, v0, v1, priv);
        WAIT_V(8);
        w_read<1, 0>(tb);
        WAIT_L0();
        w_stage(wsrc + (size_t)192 * GN, v0, v1, priv + 8192);
        STAMP(2);
        WAIT_V(8);
        w_read<2, 0>(tb);
        WAIT_V(0);
        w_read<3, 0>(tb);
        STAMP(3);
    }
    unsigned call;
    asm volatile("v_readfirstlane_b32 %0, v%c1" : "=s"(call) : "n"(R_POLL) : "memory");
    call += 1;

    const unsigned char* xbase = myws + WS_X + (call & 1) * X_PAR + peer * X_SRC + (isB ? 32 * 4096 : 0);
    const unsigned toff = ((2 * wid) * GN + nt * 128) * 4;
    unsigned char* const pbase = myws + WS_P + toff;
    const unsigned char* pload = pbase;
    unsigned char* const obase = (unsigned char*)p.out + toff;
    const unsigned pf0 = WS_PF + nt * 32 * 128;
    const unsigned ooff = (lane >> 5) * GN * 4 + (lane & 31) * 16;
    const unsigned wa0 = priv + m * 512 + ((g ^ (m & 7)) * 16);
    const int rrow = 2 * wid + (lane >> 5);
    const unsigned ra0 = lds0 + rrow * 512 + (((lane & 31) ^ (rrow & 7)) * 16);
    unsigned pa = lane < 16 ? WS_XF + (peer * 8 + cls) * 128 + lane * 4 : WS_TMO;
    const int first = isB ? 32 - ROT_S : ROT_S;
    const unsigned np = isB ? (unsigned)(first - sub + 2) / 3u : (unsigned)(first - sub + 3) / 4u;
    unsigned rb0 = (sub + rot) & 31, l16 = lane16;
    unsigned si, rbn, rb1, rb2, rb3, t0, t1;
    unsigned long long ret, tm0, tm1, sel;
    asm volatile(STREAM_ASM
        : [i] "=&s"(si), [rbn] "=&s"(rbn), [rb0] "+&s"(rb0), [rb1] "=&s"(rb1), [rb2] "=&s"(rb2), [rb3] "=&s"(rb3),
          [t0] "=&s"(t0), [t1] "=&s"(t1), [ret] "=&s"(ret), [tm0] "=&s"(tm0), [tm1] "=&s"(tm1), [sel] "=&s"(sel),
          [xbase] "+&s"(xbase), [pload] "+&s"(pload), [pa] "+&v"(pa), [l16] "+&v"(l16)
        : [pbase] "s"(pbase), [obase] "s"(obase), [myws] "s"(myws), [dummy] "s"(p.w), [call] "s"(call),
          [nblk] "s"(nblk), [np] "s"(np), [stride] "s"(stride), [pf0] "s"(pf0), [npub] "s"(wid == 0 ? np : 0u),
          [cnt] "s"(WS_CNT + b * 128), [icnt] "s"(wid * 8),
          [lastbit] "s"(1u << nblk), [ooff] "v"(ooff), [wa0] "v"(wa0), [wa1] "v"(wa0 ^ 64),
          [ra0] "v"(ra0), [ra1] "v"(ra0 + 65536), [vcall] "v"(call), [vzero] "v"(0u), [tmo] "n"(WS_TMO), [ddl] "n"(DEADLINE_TICKS)
        : "memory", "vcc", "scc");
    DIAG_DUMP();
}

extern "C" void kernel_launch(int rank, int nranks, ncclComm_t comm,
                              hipStream_t stream,
                              void* const* d_in, const int* in_sizes, int n_in,
                              void* d_out, int out_size,
                              const int* in_dtypes, int out_dtype,
                              void* const* d_out_all,
                              void* const* d_ws_all, size_t ws_bytes) {
    (void)comm; (void)in_dtypes; (void)out_dtype; (void)d_out_all;
    if (nranks != NR || (unsigned)rank >= (unsigned)NR || n_in != 4 || ws_bytes < WS_END ||
        in_sizes[0] != 4096 * KS || in_sizes[1] != 4096 * GN || out_size != ML * GN) {
        fprintf(stderr, "unexpected problem shape\n");
        return;
    }
    HIP_CHECK(hipFuncSetAttribute((const void*)a2a_gemm, hipFuncAttributeMaxDynamicSharedMemorySize, L_END));
    Params p;
    p.x = (const unsigned char*)d_in[0];
    p.w = (const unsigned char*)d_in[1];
    p.sx = (const float*)d_in[2];
    p.sw = (const float*)d_in[3];
    p.out = (float*)d_out;
    p.myws = (unsigned char*)d_ws_all[rank];
    for (int i = 0; i < NR; ++i) p.ws[i] = (unsigned char*)d_ws_all[(rank + i) % NR];
    p.rank = rank;
    p.pad = 0;
    int pa = 0, pb = 0;
    for (int s = 0; s < 64; ++s) {
        const bool takeB = pa >= 32 || (pb < 32 && (pb / 3) * 460 <= A_LAG + (pa / 4) * 460);
        unsigned u;
        if (takeB) { u = 1u | (((pb + ROT_S) & 31) << 1) | ((pb % 3) << 6) | ((pb / 3) << 9); ++pb; }
        else       { u = 0u | (pa << 1) | ((3 + pa % 4) << 6) | ((pa / 4) << 9); ++pa; }
        if (s < NPUSH) p.sched[s] = u; else p.sched[s - NPUSH] |= u << 16;
    }
    a2a_gemm<<<NWG, NTHREADS, L_END, stream>>>(p);
    HIP_CHECK(hipGetLastError());
}
```

```cpp
#include <hip/hip_runtime.h>
#include <rccl/rccl.h>
#include <cstdio>
#include <cstdint>
#include <cstddef>

#pragma clang diagnostic ignored "-Winline-asm"

#define HIP_CHECK(expr) do { hipError_t _e = (expr); if (_e != hipSuccess) { \
    fprintf(stderr, "HIP error %s at %s:%d\n", hipGetErrorString(_e), __FILE__, __LINE__); \
    return; } } while (0)

constexpr int NR = 8;
constexpr int GN = 4096;
constexpr int KS = 512;
constexpr int ML = 512;
constexpr int NWG = 256, NTHREADS = 512, NPUSH = 32;

#define ROT_S 14
#define SLOT_T0 90
#define SLOT_DT 74
#define A_LAG 1400
#define A_DELAY 0
#define B_DELAY 0
#define SPIN_SLEEP 4
#define LATE2 1
#define ROTD 0
#define PREF 1
#define NTMAP 3
#define STAGGER 3
static_assert(ROT_S >= 9 && ROT_S <= 20, "the last three blocks of every CU must be blocks it owns");

#define WS_TMO  8192
#define WS_XF   16384
#define WS_CNT  32768
#define WS_PF   65536
#define WS_X    262144
#define WS_P    (WS_X + 8 * 2097152)
#define WS_END  (WS_P + 4 * 8388608)
#define DEADLINE_TICKS 20000000
constexpr unsigned L_END = 8 * 16384;

struct Params {
    const void* x;
    const void* w;
    const void* sx;
    const void* sw;
    void* out;
    void* myws;
    unsigned rank;
    unsigned pad[3];
    void* ws[NR];
    unsigned sched[NPUSH][2];
};
static_assert(offsetof(Params, ws) == 64 && offsetof(Params, sched) == 128, "kernarg layout");

#define STR2(x) #x
#define STR(x) STR2(x)
#define ABL 0
#define DIAG_ON 0

#define BODY_ASM \
    ".set ROTD, " STR(ROTD) "\n .set PREF, " STR(PREF) "\n .set NTMAP, " STR(NTMAP) "\n .set ABL, " STR(ABL) "\n .set DIAG, " STR(DIAG_ON) "\n .set ROT_S, " STR(ROT_S) "\n" \
    ".set s_wid, 34\n .set s_call, 35\n .set s_t0, 38\n .set s_t1, 39\n .set s_t2, 40\n .set s_t3, 41\n" \
    ".set s_priv, 42\n .set s_peer, 43\n .set s_cls, 44\n .set s_nt, 45\n .set s_nblk, 46\n .set s_np, 47\n" \
    ".set s_i, 48\n .set s_want, 49\n .set s_notv, 50\n .set s_spec, 51\n .set s_strx, 52\n .set s_stro, 53\n" \
    ".set s_scale, 54\n .set s_mul, 55\n .set s_pmask, 56\n .set s_np2, 57\n .set s_rb0, 58\n .set s_stride, 59\n" \
    ".set s_xlo, 60\n .set s_xhi, 62\n .set s_pst, 64\n .set s_pld, 66\n .set s_ob, 68\n .set s_sel, 70\n" \
    ".set s_ret, 72\n .set s_tm0, 74\n .set s_tm1, 76\n .set s_src, 78\n .set s_b, 80\n .set s_isb, 81\n" \
    ".set s_d0, 82\n .set s_d2, 84\n .set s_nb2, 85\n" \
    ".macro STAMP idx\n .if DIAG\n" \
    " s_memrealtime s[s_d0:s_d0+1]\n s_lshl_b32 s[s_d2], s[s_wid], 5\n s_add_u32 s[s_d2], s[s_d2], \\idx\n" \
    " s_lshl_b32 s[s_d2], s[s_d2], 2\n s_add_u32 s[s_d2], s[s_d2], 131072\n v_mov_b32 v18, s[s_d2]\n" \
    " s_waitcnt lgkmcnt(0)\n v_mov_b32 v19, s[s_d0]\n ds_write_b32 v18, v19\n .endif\n.endm\n" \
    ".macro CSTAMP idx\n .if DIAG\n" \
    " s_memtime s[s_d0:s_d0+1]\n s_lshl_b32 s[s_d2], s[s_wid], 5\n s_add_u32 s[s_d2], s[s_d2], \\idx\n" \
    " s_lshl_b32 s[s_d2], s[s_d2], 2\n s_add_u32 s[s_d2], s[s_d2], 131072\n v_mov_b32 v18, s[s_d2]\n" \
    " s_waitcnt lgkmcnt(0)\n v_mov_b32 v19, s[s_d0]\n ds_write_b32 v18, v19\n .endif\n.endm\n" \
    ".macro DUMP\n .if DIAG\n" \
    " s_waitcnt vmcnt(0)\n STAMP 30\n v_and_b32 v18, 0x1f0, v8\n v_lshrrev_b32 v18, 2, v18\n" \
    " s_lshl_b32 s[s_t0], s[s_wid], 7\n s_add_u32 s[s_t1], s[s_t0], 131072\n v_add_u32 v19, s[s_t1], v18\n ds_read_b32 v19, v19\n" \
    " s_lshl_b32 s[s_t1], s[s_b], 10\n s_add_u32 s[s_t0], s[s_t0], s[s_t1]\n s_add_u32 s[s_t0], s[s_t0], " STR(WS_END) "\n" \
    " v_add_u32 v18, s[s_t0], v18\n s_waitcnt lgkmcnt(0)\n global_store_dword v18, v19, s[26:27]\n .endif\n.endm\n" \
    ".macro WORD base\n"                   \
    " s_mov_b64 exec, 1\n global_store_dword v21, v17, s[\\base:\\base+1] sc0 sc1\n s_mov_b64 exec, -1\n.endm\n" \
    ".macro DMA v, step\n"                 \
    " global_load_lds_dwordx4 v\\v, s[s_src:s_src+1]\n s_add_u32 m0, m0, 1024\n" \
    " s_add_u32 s[s_src], s[s_src], \\step\n s_addc_u32 s[s_src+1], s[s_src+1], 0\n.endm\n" \
    ".macro UNTIL t\n"                     \
    "1:\n s_memrealtime s[s_tm0:s_tm0+1]\n s_waitcnt lgkmcnt(0)\n s_sub_u32 s[s_tm0], s[s_tm0], s[\\t]\n" \
    " s_cmp_lt_i32 s[s_tm0], 0\n s_cbranch_scc0 2f\n s_sleep 1\n s_branch 1b\n2:\n.endm\n" \
    ".macro WREAD q\n"                     \
    " .irp c, 0,1,2,3,4,5,6,7\n" \
    " ds_read_b64_tr_b8 v[128+((\\q>>1)*8+\\c)*8+(\\q&1)*4:129+((\\q>>1)*8+\\c)*8+(\\q&1)*4], v[68+\\c] offset:(\\q&1)*8192\n" \
    " ds_read_b64_tr_b8 v[130+((\\q>>1)*8+\\c)*8+(\\q&1)*4:131+((\\q>>1)*8+\\c)*8+(\\q&1)*4], v[68+\\c] offset:(\\q&1)*8192+1024\n" \
    " .endr\n.endm\n" \
    ".macro MM0 c, x\n .if (ABL & 256)\n .exitm\n .endif\n v_mfma_f32_16x16x128_f8f6f4 v[96+4*\\c:99+4*\\c], v[128+8*\\c:135+8*\\c], v[\\x:\\x+7], 0\n.endm\n" \
    ".macro MM1 c, x\n .if (ABL & 256)\n .exitm\n .endif\n v_mfma_f32_16x16x128_f8f6f4 v[96+4*\\c:99+4*\\c], v[192+8*\\c:199+8*\\c], v[\\x+8:\\x+15], v[96+4*\\c:99+4*\\c]\n.endm\n" \
    ".macro WR c, slot\n .if (ABL & 512)\n .irp e, 0,1,2,3\n ds_write_addtid_b32 v[96+4*\\c+\\e] offset:\\slot+\\c*1024+\\e*256\n .endr\n" \
    " .elseif (ABL & 1024)\n ds_write_b64 v8, v[96+4*\\c:97+4*\\c] offset:\\slot+\\c*1024\n ds_write_b64 v8, v[98+4*\\c:99+4*\\c] offset:\\slot+\\c*1024+8\n" \
    " .elseif !(ABL & 1)\n ds_write_b128 v[12+(\\c&1)], v[96+4*\\c:99+4*\\c] offset:\\slot+(\\c>>1)*128\n .endif\n.endm\n" \
    ".macro RD w, slot\n .if !(ABL & 2)\n ds_read_b128 v[64+4*\\w:67+4*\\w], v[14+(\\w>>2)] offset:\\slot+(\\w&3)*16384\n .endif\n.endm\n" \
    ".macro ADD4 d, a, b\n .if !(ABL & 4)\n v_add_f32 v[\\d], v[\\a], v[\\b]\n v_add_f32 v[\\d+1], v[\\a+1], v[\\b+1]\n" \
    " v_add_f32 v[\\d+2], v[\\a+2], v[\\b+2]\n v_add_f32 v[\\d+3], v[\\a+3], v[\\b+3]\n .endif\n.endm\n" \
    ".macro MUL4 s\n v_mul_f32 v24, s[\\s], v24\n v_mul_f32 v25, s[\\s], v25\n v_mul_f32 v26, s[\\s], v26\n v_mul_f32 v27, s[\\s], v27\n.endm\n" \
    ".macro LX x, u\n .if (ABL & 32)\n global_load_dword v22, v16, s[26:27]\n .exitm\n .endif\n global_load_dwordx4 v[\\x+4*\\u:\\x+4*\\u+3], v9, s[s_xlo+(\\u&2):s_xlo+(\\u&2)+1] offset:1024*(\\u&1) sc0 sc1\n.endm\n" \
    ".macro POLL\n global_load_dword v20, v16, s[26:27] sc0 sc1\n.endm\n" \
    ".macro STORE vs\n .if (ABL & 128)\n global_load_dword v22, v16, s[26:27]\n .exitm\n .endif\n global_store_dwordx4 v\\vs, v[24:27], s[s_sel:s_sel+1] sc0 sc1\n.endm\n" \
      \
    ".macro ITER id, xc, xn, sc, sp, vs, pb\n" \
    " s_waitcnt vmcnt(7)\n" \
    " .if (DIAG == 1)\n s_add_u32 s[s_t0], s[s_i], 4\n STAMP s[s_t0]\n .endif\n" \
    " .if (DIAG == 2)\n s_cmp_eq_u32 s[s_i], 2\n s_cbranch_scc0 9f\n STAMP 4\n CSTAMP 16\n9:\n .endif\n" \
    " MM0 0, \\xc\n" \
    " WR 4, \\sp\n" \
    " LX \\xn, 2\n" \
    " s_cmp_lt_u32 s[s_i], s[s_np2]\n s_cselect_b64 s[s_sel:s_sel+1], s[s_pst:s_pst+1], s[s_ob:s_ob+1]\n" \
    " s_cselect_b32 s[s_mul], 1.0, s[s_scale]\n" \
    " MM0 1, \\xc\n" \
    " LX \\xn, 3\n" \
    " ADD4 24, 24, \\pb\n" \
    " MM0 2, \\xc\n" \
    " WR 5, \\sp\n" \
    " MUL4 s_mul\n" \
    " s_cmp_lt_u32 s[s_i], 2\n" \
    " MM0 3, \\xc\n" \
    " s_cbranch_scc1 .La2a_nost\\id\n" \
    " STORE \\vs\n" \
    ".La2a_st\\id:\n" \
    " MM0 4, \\xc\n" \
    " WR 6, \\sp\n" \
    " POLL\n" \
    " MM0 5, \\xc\n" \
    " WR 7, \\sp\n" \
    " v_add_u32 v\\vs, s[s_stro], v\\vs\n v_add_u32 v9, s[s_strx], v9\n" \
    " s_cmp_lt_u32 s[s_i], s[s_nb2]\n s_cselect_b32 s[s_t1], 0x1ffff, 0\n" \
    " MM0 6, \\xc\n" \
    " v_and_b32 v\\vs, 0x7fffff, v\\vs\n v_and_b32 v9, s[s_t1], v9\n" \
    " s_cmp_ge_u32 s[s_i], s[s_np]\n s_cselect_b32 s[s_pmask], -1, 0\n" \
    " MM0 7, \\xc\n" \
    " s_waitcnt vmcnt(7) lgkmcnt(0)\n" \
    " v_cmp_le_u32 vcc, s[s_call], v20\n" \
    " .if !(ABL & 16)\n s_barrier\n .endif\n" \
    " MM1 0, \\xc\n" \
    " RD 0, \\sp\n RD 1, \\sp\n" \
    " s_or_b32 s[s_t0], vcc_lo, s[s_notv]\n s_andn2_b32 s[s_t0], s[s_want], s[s_t0]\n" \
    " s_cbranch_scc1 .La2a_slow\\id\n" \
    ".La2a_go\\id:\n" \
    " MM1 1, \\xc\n" \
    " RD 2, \\sp\n RD 3, \\sp\n" \
    " LX \\xc, 0\n" \
    " MM1 2, \\xc\n" \
    " RD 4, \\sp\n RD 5, \\sp\n WR 0, \\sc\n" \
    " LX \\xc, 1\n" \
    " s_bitcmp1_b32 s[s_spec], s[s_i]\n" \
    " MM1 3, \\xc\n" \
    " RD 6, \\sp\n RD 7, \\sp\n" \
    " s_cbranch_scc1 .La2a_spec\\id\n" \
    ".La2a_p\\id:\n" \
    " s_cmp_ge_u32 s[s_i], s[s_np]\n s_cselect_b64 s[s_sel:s_sel+1], s[s_pld:s_pld+1], s[26:27]\n" \
    " v_and_b32 v21, s[s_pmask], v\\vs\n" \
    " global_load_dwordx4 v[\\pb:\\pb+3], v21, s[s_sel:s_sel+1] sc0 sc1\n" \
    ".La2a_q\\id:\n" \
    " .if (ABL & 512)\n s_waitcnt lgkmcnt(8)\n .elseif (ABL & 1024)\n s_waitcnt lgkmcnt(6)\n .else\n s_waitcnt lgkmcnt(5)\n .endif\n" \
    " ADD4 64, 64, 68\n" \
    " MM1 4, \\xc\n" \
    " WR 1, \\sc\n" \
    " ADD4 72, 72, 76\n ADD4 64, 64, 72\n" \
    " MM1 5, \\xc\n" \
    " .if (ABL & 512)\n s_waitcnt lgkmcnt(4)\n .elseif (ABL & 1024)\n s_waitcnt lgkmcnt(2)\n .else\n s_waitcnt lgkmcnt(1)\n .endif\n" \
    " WR 2, \\sc\n" \
    " ADD4 80, 80, 84\n" \
    " s_lshl_b32 s[s_want], s[s_want], 1\n s_add_u32 s[s_i], s[s_i], 1\n" \
    " MM1 6, \\xc\n" \
    " ADD4 88, 88, 92\n ADD4 80, 80, 88\n" \
    " MM1 7, \\xc\n" \
    " WR 3, \\sc\n" \
    " ADD4 24, 64, 80\n" \
    " s_cmp_lt_u32 s[s_i], s[s_nblk]\n" \
    ".endm\n" \
    ".macro SIDE id\n"                     \
    ".La2a_slow\\id:\n s_call_b64 s[s_ret:s_ret+1], .La2a_spin\n s_branch .La2a_go\\id\n" \
    ".La2a_nost\\id:\n global_load_dword v22, v16, s[26:27]\n s_branch .La2a_st\\id\n"        \
    ".La2a_spec\\id:\n s_call_b64 s[s_ret:s_ret+1], .La2a_pub\n" \
    " s_cmp_ge_u32 s[s_i], s[s_np]\n s_cbranch_scc1 .La2a_p\\id\n s_branch .La2a_q\\id\n" \
    ".endm\n" \
    ".macro DRAIN sp, vs, pb, vs1, pb1\n"   \
    " .if (DIAG == 2)\n STAMP 5\n CSTAMP 17\n .endif\n" \
    " WR 4, \\sp\n WR 5, \\sp\n" \
    " s_mov_b64 s[s_sel:s_sel+1], s[s_ob:s_ob+1]\n" \
    " s_waitcnt vmcnt(7)\n" \
    " ADD4 24, 24, \\pb\n" \
    " WR 6, \\sp\n WR 7, \\sp\n" \
    " MUL4 s_scale\n STORE \\vs\n" \
    " s_waitcnt lgkmcnt(0)\n s_barrier\n" \
    " RD 0, \\sp\n RD 1, \\sp\n RD 2, \\sp\n RD 3, \\sp\n RD 4, \\sp\n RD 5, \\sp\n RD 6, \\sp\n RD 7, \\sp\n" \
    " s_waitcnt lgkmcnt(4)\n" \
    " ADD4 64, 64, 68\n ADD4 72, 72, 76\n ADD4 64, 64, 72\n" \
    " s_waitcnt lgkmcnt(0)\n" \
    " ADD4 80, 80, 84\n ADD4 88, 88, 92\n ADD4 80, 80, 88\n ADD4 24, 64, 80\n" \
    " s_waitcnt vmcnt(1)\n" \
    " ADD4 24, 24, \\pb1\n MUL4 s_scale\n STORE \\vs1\n" \
    " STAMP 28\n DUMP\n s_endpgm\n" \
    ".endm\n" \
      \
    " s_memrealtime s[36:37]\n" \
    " s_load_dwordx16 s[16:31], %[ka], 0\n" \
    " v_readfirstlane_b32 s[s_wid], %[tid]\n" \
    " s_mov_b32 s[s_b], %[bid]\n s_lshr_b32 s[s_wid], s[s_wid], 6\n" \
    " s_and_b32 s[s_t0], %[bid], 24\n s_lshl_b32 s[s_t1], s[s_wid], 5\n s_add_u32 s[s_t0], s[s_t0], s[s_t1]\n" \
    " s_load_dwordx2 s[32:33], %[ka], s[s_t0] offset:128\n"                  \
    " s_mul_i32 s[s_peer], s[s_wid], ROTD\n s_add_u32 s[s_peer], s[s_peer], %[bid]\n s_and_b32 s[s_peer], s[s_peer], 7\n" \
    " s_lshl_b32 s[s_t1], s[s_peer], 3\n" \
    " s_load_dwordx2 s[s_sel:s_sel+1], %[ka], s[s_t1] offset:64\n" \
    " v_and_b32 v23, 63, %[tid]\n v_lshlrev_b32 v8, 4, v23\n"                 \
    " v_lshrrev_b32 v22, 4, v23\n v_and_b32 v21, 15, v23\n" \
    " s_lshl_b32 s[s_priv], s[s_wid], 14\n s_mov_b32 m0, s[s_priv]\n" \
    " s_lshl_b32 s[s_t0], s[s_b], 7\n s_add_u32 s[s_t0], s[s_t0], " STR(WS_CNT) "\n v_mov_b32 v20, s[s_t0]\n" \
    " .if DIAG\n v_lshrrev_b32 v18, 2, v8\n v_and_b32 v18, 0x7c, v18\n s_lshl_b32 s[s_t0], s[s_wid], 7\n s_add_u32 s[s_t0], s[s_t0], 131072\n" \
    " v_add_u32 v18, s[s_t0], v18\n v_mov_b32 v19, 0\n ds_write_b32 v18, v19\n v_mov_b32 v18, s[s_t0]\n .endif\n" \
    " s_cmp_lt_u32 s[s_b], 32\n" \
    " s_waitcnt lgkmcnt(0)\n" \
    " .if DIAG\n v_mov_b32 v19, s36\n ds_write_b32 v18, v19\n .endif\n" \
    " s_cbranch_scc0 .La2a_compute\n" \
    " .if (ABL & 4096)\n DUMP\n s_endpgm\n .endif\n" \
      \
    " s_add_u32 s[s_peer], s28, s[s_peer]\n s_and_b32 s[s_peer], s[s_peer], 7\n" \
    " v_xor_b32 v9, v21, v22\n v_lshlrev_b32 v9, 4, v9\n"                     \
    " v_lshl_add_u32 v64, v22, 9, v9\n v_xor_b32 v65, 64, v64\n v_xor_b32 v66, 0x80, v64\n v_xor_b32 v67, 0xc0, v64\n" \
    " s_lshl_b32 s[s_t3], s[s_peer], 18\n" \
    ".macro PLOAD e\n" \
    " s_bfe_u32 s[s_t0], s[32+\\e], 0x50001\n s_lshl_b32 s[s_t0], s[s_t0], 13\n s_and_b32 s[s_t1], s[32+\\e], 1\n" \
    " s_lshl_b32 s[s_t1], s[s_t1], 8\n s_add_u32 s[s_t0], s[s_t0], s[s_t1]\n s_add_u32 s[s_t0], s[s_t0], s[s_t3]\n" \
    " s_add_u32 s[s_src], s16, s[s_t0]\n s_addc_u32 s[s_src+1], s17, 0\n" \
    " DMA 64, 2048\n DMA 65, 2048\n DMA 66, 2048\n DMA 67, 2048\n.endm\n" \
    " PLOAD 0\n global_load_dword v20, v20, s[26:27] sc1\n" \
    " .if !" STR(LATE2) "\n s_barrier\n PLOAD 1\n .endif\n" \
    " v_lshl_add_u32 v10, v21, 8, v9\n v_add_u32 v10, s[s_priv], v10\n"       \
    " v_xor_b32 v11, 64, v10\n v_xor_b32 v12, 0x80, v10\n v_xor_b32 v13, 0xc0, v10\n" \
    " s_lshl_b32 s[s_t0], s28, 18\n s_add_u32 s[s_t0], s[s_t0], " STR(WS_X) "\n" \
    " s_add_u32 s[s_ob], s[s_sel], s[s_t0]\n s_addc_u32 s[s_ob+1], s[s_sel+1], 0\n" \
    " s_lshl_b32 s[s_t2], s28, 10\n s_add_u32 s[s_t2], s[s_t2], " STR(WS_XF) "\n" \
    " .if " STR(LATE2) "\n s_waitcnt vmcnt(0)\n .else\n s_waitcnt vmcnt(4)\n .endif\n" \
    " v_add_u32 v17, 1, v20\n" \
    " v_and_b32 v9, 7, v17\n v_lshlrev_b32 v9, 21, v9\n v_add_u32 v8, v8, v9\n"     \
    " STAMP 1\n" \
    " .irp e, 0,1\n" \
    " ds_read_b128 v[32:35], v10 offset:\\e*4096\n ds_read_b128 v[36:39], v11 offset:\\e*4096\n" \
    " ds_read_b128 v[40:43], v12 offset:\\e*4096\n ds_read_b128 v[44:47], v13 offset:\\e*4096\n" \
    " s_bfe_u32 s[s_t0], s[32+\\e], 0x50001\n s_and_b32 s[s_t1], s[32+\\e], 1\n s_lshl_b32 s[s_t1], s[s_t1], 5\n" \
    " s_add_u32 s[s_t0], s[s_t0], s[s_t1]\n s_lshl_b32 s[s_t0], s[s_t0], 12\n v_add_u32 v9, s[s_t0], v8\n" \
    " s_bfe_u32 s[s_t0], s[32+\\e], 0xa0006\n s_add_u32 s[s_t0], s[s_t0], s[s_t2]\n v_mov_b32 v21, s[s_t0]\n" \
    " s_lshr_b32 s[s_t0], s[32+\\e], 20\n s_add_u32 s[s_t0], s[s_t0], s36\n" \
    " UNTIL s_t0\n" \
    " global_store_dwordx4 v9, v[32:35], s[s_ob:s_ob+1] sc0 sc1\n" \
    " global_store_dwordx4 v9, v[36:39], s[s_ob:s_ob+1] offset:1024 sc0 sc1\n" \
    " global_store_dwordx4 v9, v[40:43], s[s_ob:s_ob+1] offset:2048 sc0 sc1\n" \
    " global_store_dwordx4 v9, v[44:47], s[s_ob:s_ob+1] offset:3072 sc0 sc1\n" \
    " STAMP 2+2*\\e\n" \
    " s_waitcnt vmcnt(0)\n" \
    " WORD s_sel\n" \
    " STAMP 3+2*\\e\n" \
    " .if (" STR(LATE2) " && \\e == 0)\n PLOAD 1\n s_waitcnt vmcnt(0)\n .endif\n" \
    " .if (\\e == 1)\n s_barrier\n s_cmp_eq_u32 s[s_wid], 0\n s_cbranch_scc0 6f\n"     \
    " s_lshl_b32 s[s_t0], s[s_b], 7\n s_add_u32 s[s_t0], s[s_t0], " STR(WS_CNT) "\n v_mov_b32 v21, s[s_t0]\n WORD 26\n6:\n .endif\n" \
    " .endr\n" \
    " .if DIAG\n v_and_b32 v8, 0x3ff, v8\n .endif\n" \
    " DUMP\n s_endpgm\n" \
      \
    ".La2a_stage:\n"                       \
    " DMA 64, 32768\n DMA 64, 32768\n.La2a_stage6:\n DMA 65, 32768\n DMA 65, 32768\n" \
    " DMA 64, 32768\n DMA 64, 32768\n DMA 65, 32768\n DMA 65, 32768\n" \
    " s_setpc_b64 s[s_ret:s_ret+1]\n" \
    ".La2a_compute:\n" \
    " s_and_b32 s[s_nt], s[s_b], 31\n" \
    " .if (NTMAP == 0)\n s_and_b32 s[s_t0], s[s_b], 7\n s_lshl_b32 s[s_t0], s[s_t0], 2\n s_lshr_b32 s[s_nt], s[s_nt], 3\n s_add_u32 s[s_nt], s[s_nt], s[s_t0]\n .endif\n" \
    " .if (NTMAP == 2)\n s_and_b32 s[s_t0], s[s_b], 7\n s_lshl_b32 s[s_t0], s[s_t0], 1\n s_bfe_u32 s[s_t1], s[s_b], 0x10003\n s_and_b32 s[s_nt], s[s_nt], 16\n" \
    " s_add_u32 s[s_nt], s[s_nt], s[s_t0]\n s_add_u32 s[s_nt], s[s_nt], s[s_t1]\n .endif\n" \
    " .if (NTMAP == 3)\n s_lshr_b32 s[s_t0], s[s_nt], 3\n s_add_u32 s[s_t0], s[s_t0], s[s_nt]\n s_and_b32 s[s_t0], s[s_t0], 7\n s_and_b32 s[s_nt], s[s_nt], 24\n s_or_b32 s[s_nt], s[s_nt], s[s_t0]\n .endif\n" \
    " s_lshr_b32 s[s_cls], s[s_b], 5\n s_sub_u32 s[s_cls], s[s_cls], 1\n" \
    " s_sub_u32 s[s_t3], s[s_cls], 3\n s_cmp_lt_u32 s[s_cls], 3\n" \
    " s_cselect_b32 s[s_isb], 1, 0\n s_cselect_b32 s[s_t3], s[s_cls], s[s_t3]\n"            \
    " s_cselect_b32 s[s_stride], 3, 4\n s_cselect_b32 s[s_rb0], ROT_S, 0\n" \
    " s_cselect_b32 s[s_t2], 0, " STR(A_DELAY) "\n" \
    " s_add_u32 s[s_rb0], s[s_rb0], s[s_t3]\n" \
    " s_and_b32 s[s_pmask], s[s_t3], " STR(STAGGER) "\n"                                   \
    " s_lshl_b32 s[s_t0], s[s_pmask], 2\n s_add_u32 s[s_peer], s28, s[s_wid]\n s_add_u32 s[s_peer], s[s_peer], s[s_t0]\n" \
    " s_and_b32 s[s_peer], s[s_peer], 7\n s_lshr_b32 s[s_pmask], s[s_pmask], 1\n" \
    " s_lshl_b32 s[s_t0], s[s_peer], 21\n s_lshl_b32 s[s_t1], s[s_isb], 20\n s_add_u32 s[s_t0], s[s_t0], s[s_t1]\n" \
    " s_lshl_b32 s[s_t1], s[s_nt], 7\n s_add_u32 s[s_t0], s[s_t0], s[s_t1]\n" \
    " s_lshl_b32 s[s_t1], s[s_pmask], 19\n s_add_u32 s[s_mul], s[s_t0], s[s_t1]\n"         \
    " v_and_b32 v9, 7, v23\n v_xor_b32 v9, v9, v22\n v_lshlrev_b32 v9, 4, v9\n"            \
    " v_lshrrev_b32 v10, 3, v23\n v_lshl_add_u32 v64, v10, 12, v9\n v_xor_b32 v65, 64, v64\n" \
    " global_load_dword v20, v20, s[26:27] sc1\n" \
    " .if " STR(B_DELAY) "\n s_add_u32 s[s_t1], s36, " STR(B_DELAY) "\n UNTIL s_t1\n .endif\n" \
      \
    " .if PREF\n s_cmp_eq_u32 s[s_isb], 1\n s_cbranch_scc1 3f\n" \
    " s_mul_i32 s[s_t1], s[s_t3], 48*4096\n s_add_u32 s[s_t0], s[s_t0], s[s_t1]\n s_add_u32 s[s_t0], s[s_t0], (256+64)*4096\n" \
    " s_add_u32 s[s_src], s18, s[s_t0]\n s_addc_u32 s[s_src+1], s19, 0\n" \
    " s_call_b64 s[s_ret:s_ret+1], .La2a_stage6\n s_mov_b32 m0, s[s_priv]\n s_barrier\n3:\n .endif\n" \
    " s_add_u32 s[s_src], s18, s[s_mul]\n s_addc_u32 s[s_src+1], s19, 0\n" \
    " s_cmp_eq_u32 s[s_t2], 0\n s_cbranch_scc1 2f\n s_add_u32 s[s_t2], s[s_t2], s36\n UNTIL s_t2\n" \
    " s_call_b64 s[s_ret:s_ret+1], .La2a_stage\n" \
    " s_call_b64 s[s_ret:s_ret+1], .La2a_stage\n" \
    " STAMP 1\n" \
    " s_load_dword s32, s[20:21], 0\n s_load_dword s33, s[22:23], 0\n" \
      \
    " v_lshl_add_u32 v12, v21, 9, v9\n v_add_u32 v12, s[s_priv], v12\n v_xor_b32 v13, 64, v12\n" \
    " v_lshrrev_b32 v10, 1, v21\n v_lshl_add_u32 v11, v22, 4, v10\n v_lshlrev_b32 v11, 7, v11\n"     \
    " v_lshrrev_b32 v10, 1, v10\n v_and_b32 v14, 1, v22\n v_lshl_add_u32 v10, v14, 2, v10\n v_lshl_add_u32 v11, v10, 4, v11\n" \
    " v_and_b32 v10, 1, v23\n v_lshl_add_u32 v11, v10, 3, v11\n v_add_u32 v68, s[s_priv], v11\n" \
    " .irp c, 1,2,3,4,5,6,7\n v_xor_b32 v[68+\\c], 16*\\c, v68\n .endr\n" \
    " v_lshrrev_b32 v10, 5, v23\n v_and_b32 v11, 31, v23\n s_lshl_b32 s[s_t0], s[s_wid], 1\n" \
    " v_add_u32 v14, s[s_t0], v10\n v_and_b32 v15, 7, v14\n v_xor_b32 v15, v15, v11\n v_lshlrev_b32 v15, 4, v15\n" \
    " v_lshl_add_u32 v14, v14, 9, v15\n v_add_u32 v15, 0x10000, v14\n" \
    " v_lshlrev_b32 v11, 4, v11\n v_lshl_add_u32 v11, v10, 14, v11\n"                       \
    " s_lshl_b32 s[s_strx], s[s_stride], 12\n s_lshl_b32 s[s_stro], s[s_stride], 19\n" \
    " s_lshl_b32 s[s_t0], s[s_rb0], 18\n s_sub_u32 s[s_t0], s[s_t0], s[s_stro]\n s_and_b32 s[s_t1], s[s_t0], 0x7fffff\n" \
    " v_add_u32 v10, s[s_t1], v11\n s_lshr_b32 s[s_t1], s[s_stro], 1\n s_add_u32 s[s_t0], s[s_t0], s[s_t1]\n" \
    " s_and_b32 s[s_t0], s[s_t0], 0x7fffff\n v_add_u32 v11, s[s_t0], v11\n" \
    " s_lshl_b32 s[s_t0], s[s_rb0], 12\n v_add_u32 v9, s[s_t0], v8\n" \
    " s_lshl_b32 s[s_t0], s[s_wid], 15\n s_lshl_b32 s[s_t1], s[s_nt], 9\n s_add_u32 s[s_t0], s[s_t0], s[s_t1]\n" \
    " s_add_u32 s[s_ob], s24, s[s_t0]\n s_addc_u32 s[s_ob+1], s25, 0\n s_add_u32 s[s_pst], s[s_t0], " STR(WS_P) "\n" \
      \
    " s_lshl_b32 s[s_t0], s[s_peer], 3\n s_add_u32 s[s_t0], s[s_t0], s[s_cls]\n s_lshl_b32 s[s_t0], s[s_t0], 7\n s_add_u32 s[s_t0], s[s_t0], " STR(WS_XF) "\n" \
    " s_lshl_b32 s[s_t1], s[s_nt], 3\n s_add_u32 s[s_t1], s[s_t1], s[s_cls]\n s_lshl_b32 s[s_t1], s[s_t1], 7\n s_add_u32 s[s_t1], s[s_t1], " STR(WS_PF) "\n" \
    " v_mov_b32 v16, " STR(WS_TMO) "\n v_lshlrev_b32 v21, 2, v21\n" \
    " s_mov_b32 exec_hi, 0\n v_add_u32 v16, s[s_t1], v21\n s_mov_b32 exec_lo, 0xffff\n v_add_u32 v16, s[s_t0], v21\n s_mov_b64 exec, -1\n" \
      \
    " s_cmp_eq_u32 s[s_cls], 2\n s_cselect_b32 s[s_nblk], 10, 11\n" \
    " s_sub_u32 s[s_t0], 34-ROT_S, s[s_t3]\n s_mul_i32 s[s_t0], s[s_t0], 43\n s_lshr_b32 s[s_t0], s[s_t0], 7\n" \
    " s_sub_u32 s[s_t1], ROT_S+3, s[s_t3]\n s_lshr_b32 s[s_t1], s[s_t1], 2\n" \
    " s_cmp_eq_u32 s[s_isb], 1\n s_cselect_b32 s[s_nblk], s[s_nblk], 8\n s_cselect_b32 s[s_np], s[s_t0], s[s_t1]\n" \
    " s_add_u32 s[s_np2], s[s_np], 2\n s_sub_u32 s[s_nb2], s[s_nblk], 2\n" \
    " s_bfm_b32 s[s_t0], s[s_nblk], 0\n s_bfm_b32 s[s_t1], s[s_np], 0\n s_andn2_b32 s[s_t2], s[s_t0], s[s_t1]\n" \
    " s_lshl_b32 s[s_t2], s[s_t2], 16\n s_nor_b32 s[s_notv], s[s_t0], s[s_t2]\n" \
    " s_lshl_b32 s[s_t1], s[s_t1], 3\n s_or_b32 s[s_t1], s[s_t1], 1\n s_cmp_eq_u32 s[s_wid], 0\n s_cselect_b32 s[s_spec], s[s_t1], 0\n" \
    " s_mov_b32 s[s_i], 0\n s_mov_b32 s[s_want], 0x10004\n" \
    " .irp c, 4,5,6,7,28,29,30,31\n v_mov_b32 v\\c, 0\n .endr\n" \
    " s_xor_b32 s[s_mul], s[s_mul], 0x80000\n s_add_u32 s[s_src], s18, s[s_mul]\n s_addc_u32 s[s_src+1], s19, 0\n" \
    " s_mov_b32 m0, s[s_priv]\n" \
    " s_waitcnt vmcnt(8)\n" \
    " WREAD 0\n" \
    " v_readfirstlane_b32 s[s_call], v20\n s_add_u32 s[s_call], s[s_call], 1\n v_mov_b32 v17, s[s_call]\n" \
    " s_and_b32 s[s_t0], s[s_call], 3\n s_lshl_b32 s[s_t0], s[s_t0], 23\n s_add_u32 s[s_t0], s[s_t0], s[s_pst]\n" \
    " s_add_u32 s[s_pst], s26, s[s_t0]\n s_addc_u32 s[s_pst+1], s27, 0\n s_mov_b64 s[s_pld:s_pld+1], s[s_pst:s_pst+1]\n" \
    " s_and_b32 s[s_t0], s[s_call], 7\n s_lshl_b32 s[s_t0], s[s_t0], 21\n s_lshl_b32 s[s_t1], s[s_peer], 18\n s_add_u32 s[s_t0], s[s_t0], s[s_t1]\n" \
    " s_lshl_b32 s[s_t1], s[s_isb], 17\n s_add_u32 s[s_t0], s[s_t0], s[s_t1]\n s_add_u32 s[s_t0], s[s_t0], " STR(WS_X) "\n" \
    " s_lshl_b32 s[s_t1], s[s_pmask], 11\n s_add_u32 s[s_t0], s[s_t0], s[s_t1]\n" \
    " s_add_u32 s[s_xlo], s26, s[s_t0]\n s_addc_u32 s[s_xlo+1], s27, 0\n s_xor_b32 s[s_t0], s[s_t0], 2048\n" \
    " s_add_u32 s[s_xhi], s26, s[s_t0]\n s_addc_u32 s[s_xhi+1], s27, 0\n" \
    " s_waitcnt lgkmcnt(0)\n" \
    " s_call_b64 s[s_ret:s_ret+1], .La2a_stage\n" \
    " v_mov_b32 v21, s32\n v_mul_f32 v21, s33, v21\n" \
    " s_waitcnt vmcnt(8)\n" \
    " WREAD 1\n" \
    " v_readfirstlane_b32 s[s_scale], v21\n" \
    " POLL\n" \
    " s_waitcnt lgkmcnt(0)\n" \
    " s_call_b64 s[s_ret:s_ret+1], .La2a_stage\n" \
    " STAMP 2\n" \
    " s_waitcnt vmcnt(8)\n" \
    " v_cmp_le_u32 vcc, s[s_call], v20\n" \
    " WREAD 2\n" \
    " .if (ABL & 64)\n s_branch 7f\n .endif\n s_andn2_b32 s[s_t0], 3, vcc_lo\n s_cbranch_scc0 7f\n" \
    " s_call_b64 s[s_ret:s_ret+1], .La2a_spin\n" \
    "7:\n" \
    " LX 32, 0\n LX 32, 1\n LX 32, 2\n LX 32, 3\n" \
    " v_add_u32 v9, s[s_strx], v9\n LX 48, 0\n LX 48, 1\n POLL\n" \
    " s_waitcnt vmcnt(7)\n" \
    " WREAD 3\n" \
    " .if (ABL & 64)\n s_mov_b32 s[s_notv], -1\n .endif\n .if (ABL & 512)\n s_lshr_b32 m0, s[s_priv], 1\n .endif\n" \
    " .if (ABL & 2048)\n s_cmp_ge_u32 s[s_wid], 4\n s_cbranch_scc0 3f\n s_setprio 1\n3:\n .endif\n" \
    " STAMP 3\n" \
    " s_waitcnt vmcnt(3) lgkmcnt(0)\n" \
    ".La2a_loop:\n" \
    " ITER 0, 32, 48, 0, 8192, 10, 28\n" \
    " s_cbranch_scc0 .La2a_drain0\n" \
    " ITER 1, 48, 32, 8192, 0, 11, 4\n" \
    " s_cbranch_scc1 .La2a_loop\n" \
    " DRAIN 8192, 10, 28, 11, 4\n" \
    ".La2a_drain0:\n" \
    " DRAIN 0, 11, 4, 10, 28\n" \
    " SIDE 0\n SIDE 1\n" \
      \
    ".La2a_pub:\n" \
    " s_lshl_b32 s[s_t0], s[s_b], 7\n s_add_u32 s[s_t0], s[s_t0], " STR(WS_CNT) "\n" \
    " s_cmp_eq_u32 s[s_i], 0\n s_cbranch_scc1 6f\n" \
    " s_sub_u32 s[s_t0], s[s_i], 3\n s_mul_i32 s[s_t0], s[s_t0], s[s_stride]\n s_add_u32 s[s_t0], s[s_t0], s[s_rb0]\n" \
    " s_lshr_b32 s[s_t1], s[s_t0], 2\n s_and_b32 s[s_t2], s[s_t0], 3\n s_add_u32 s[s_t2], s[s_t2], 3\n" \
    " s_cmp_eq_u32 s[s_isb], 1\n s_cbranch_scc1 8f\n" \
    " s_add_u32 s[s_t0], s[s_t0], 32-ROT_S\n s_mul_i32 s[s_t1], s[s_t0], 43\n s_lshr_b32 s[s_t1], s[s_t1], 7\n" \
    " s_mul_i32 s[s_t2], s[s_t1], 3\n s_sub_u32 s[s_t2], s[s_t0], s[s_t2]\n" \
    "8:\n s_lshl_b32 s[s_t0], s[s_nt], 3\n s_add_u32 s[s_t0], s[s_t0], s[s_t2]\n s_lshl_b32 s[s_t0], s[s_t0], 5\n" \
    " s_add_u32 s[s_t0], s[s_t0], s[s_t1]\n s_lshl_b32 s[s_t0], s[s_t0], 2\n s_add_u32 s[s_t0], s[s_t0], " STR(WS_PF) "\n" \
    "6:\n v_mov_b32 v21, s[s_t0]\n WORD 26\n s_setpc_b64 s[s_ret:s_ret+1]\n" \
      \
    ".La2a_spin:\n" \
    " s_memrealtime s[s_tm0:s_tm0+1]\n" \
    ".La2a_spin1:\n" \
    " POLL\n" \
    " s_waitcnt vmcnt(0)\n" \
    " v_cmp_le_u32 vcc, s[s_call], v20\n" \
    " s_andn2_b32 s[s_t1], s[s_t0], vcc_lo\n s_cbranch_scc0 .La2a_spin3\n" \
    " s_cmp_lg_u32 vcc_hi, 0\n s_cbranch_scc1 .La2a_spin2\n" \
    " s_memrealtime s[s_tm1:s_tm1+1]\n s_waitcnt lgkmcnt(0)\n" \
    " s_sub_u32 s[s_t1], s[s_tm1], s[s_tm0]\n" \
    " s_cmp_lt_u32 s[s_t1], " STR(DEADLINE_TICKS) "\n s_sleep " STR(SPIN_SLEEP) "\n s_cbranch_scc1 .La2a_spin1\n" \
    " v_mov_b32 v21, " STR(WS_TMO) "\n WORD 26\n s_waitcnt vmcnt(0)\n" \
    ".La2a_spin2:\n" \
    " s_mov_b64 s[s_xlo:s_xlo+1], s[18:19]\n s_mov_b64 s[s_xhi:s_xhi+1], s[18:19]\n s_mov_b64 s[s_pld:s_pld+1], s[18:19]\n" \
    " s_mov_b32 s[s_scale], -1\n s_waitcnt lgkmcnt(0)\n" \
    " .irp c, 0,1,2,3,4,5,6,7\n v_mov_b32 v[128+8*\\c], -1\n .endr\n" \
    ".La2a_spin3:\n" \
    " s_waitcnt lgkmcnt(0)\n" \
    " s_setpc_b64 s[s_ret:s_ret+1]\n"

__global__ void __launch_bounds__(NTHREADS, 2) __attribute__((amdgpu_num_vgpr(2))) a2a_gemm(Params p) {
    (void)p;
    asm volatile(BODY_ASM
        :: [ka] "s"(__builtin_amdgcn_kernarg_segment_ptr()), [bid] "s"(blockIdx.x), [tid] "v"(threadIdx.x)
        : "memory", "vcc", "scc", "v255",
          "s16", "s17", "s18", "s19", "s20", "s21", "s22", "s23", "s24", "s25", "s26", "s27", "s28", "s29", "s30", "s31",
          "s32", "s33", "s34", "s35", "s36", "s37", "s38", "s39", "s40", "s41", "s42", "s43", "s44", "s45", "s46", "s47",
          "s48", "s49", "s50", "s51", "s52", "s53", "s54", "s55", "s56", "s57", "s58", "s59", "s60", "s61", "s62", "s63",
          "s64", "s65", "s66", "s67", "s68", "s69", "s70", "s71", "s72", "s73", "s74", "s75", "s76", "s77", "s78", "s79",
          "s80", "s81", "s82", "s83", "s84", "s85");
}

extern "C" void kernel_launch(int rank, int nranks, ncclComm_t comm,
                              hipStream_t stream,
                              void* const* d_in, const int* in_sizes, int n_in,
                              void* d_out, int out_size,
                              const int* in_dtypes, int out_dtype,
                              void* const* d_out_all,
                              void* const* d_ws_all, size_t ws_bytes) {
    (void)comm; (void)in_dtypes; (void)out_dtype; (void)d_out_all;
    if (nranks != NR || (unsigned)rank >= (unsigned)NR || n_in != 4 || ws_bytes < (size_t)WS_END ||
        in_sizes[0] != 4096 * KS || in_sizes[1] != 4096 * GN || out_size != ML * GN) {
        fprintf(stderr, "unexpected problem shape\n");
        return;
    }
    HIP_CHECK(hipFuncSetAttribute((const void*)a2a_gemm, hipFuncAttributeMaxDynamicSharedMemorySize, L_END));
    Params p;
    p.x = d_in[0];
    p.w = d_in[1];
    p.sx = d_in[2];
    p.sw = d_in[3];
    p.out = d_out;
    p.myws = d_ws_all[rank];
    p.rank = (unsigned)rank;
    p.pad[0] = p.pad[1] = p.pad[2] = 0;
    for (int i = 0; i < NR; ++i) p.ws[i] = d_ws_all[(rank + i) % NR];
    int pa = 0, pb = 0;
    for (int s = 0; s < 64; ++s) {
        const bool takeB = pa >= 32 || (pb < 32 && (pb / 3) * 460 <= A_LAG + (pa / 4) * 460);
        unsigned half, rb, cls, idx;
        if (takeB) { half = 1; rb = (pb + ROT_S) & 31; cls = pb % 3; idx = pb / 3; ++pb; }
        else       { half = 0; rb = pa; cls = 3 + pa % 4; idx = pa / 4; ++pa; }
        p.sched[s % NPUSH][s / NPUSH] = half | (rb << 1) | ((cls * 128 + idx * 4) << 6) |
                                        ((unsigned)(SLOT_T0 + s * SLOT_DT / 10) << 20);
    }
    a2a_gemm<<<NWG, NTHREADS, L_END, stream>>>(p);
    HIP_CHECK(hipGetLastError());
}
```

```cpp
#include <hip/hip_runtime.h>
#include <rccl/rccl.h>
#include <cstdio>
#include <cstdint>
#include <cstddef>

#pragma clang diagnostic ignored "-Winline-asm"

#define HIP_CHECK(expr) do { hipError_t _e = (expr); if (_e != hipSuccess) { \
    fprintf(stderr, "HIP error %s at %s:%d\n", hipGetErrorString(_e), __FILE__, __LINE__); \
    return; } } while (0)

constexpr int NR = 8;
constexpr int GN = 4096;
constexpr int KS = 512;
constexpr int ML = 512;
constexpr int NWG = 256, NTHREADS = 512;

#define SLOT_DT 36
#define LOAD_LEAD 110
#define DEADLINE_TICKS 20000000

#define WS_TMO  8192
#define WS_CNT  32768
#define WS_XF   65536
#define WS_X    262144
#define WS_END  (WS_X + 8 * 2097152)
constexpr unsigned L_END = 3 * 32768 + 3 * 20480;

struct Params {
    const void* x;
    const void* w;
    const void* sx;
    const void* sw;
    void* out;
    void* myws;
    unsigned rank;
    unsigned w_own;
    unsigned x_own;
    unsigned rank_m1;
    void* ws[NR];
};
static_assert(offsetof(Params, ws) == 64, "kernarg layout");

#define STR2(x) #x
#define STR(x) STR2(x)

#define BODY_ASM \
    ".set WS_TMO, " STR(WS_TMO) "\n.set WS_XF, " STR(WS_XF) "\n.set WS_CNT, " STR(WS_CNT) "\n.set WS_X, " STR(WS_X) "\n.set L_X, 98304\n" \
    ".set s_wid, 34\n.set s_call, 35\n.set s_t0, 38\n.set s_t1, 39\n.set s_t2, 40\n.set s_t3, 41\n.set s_g, 42\n.set s_nt, 43\n.set s_rb0, 44\n" \
    ".set s_nb, 45\n.set s_wn, 46\n.set s_wk, 47\n.set s_r, 48\n.set s_i, 49\n.set s_q, 50\n.set s_xp, 51\n.set s_blind, 53\n.set s_pk, 54\n.set s_wl, 55\n" \
    ".set s_xl, 56\n.set s_b, 58\n.set s_fo, 59\n.set s_wb, 60\n.set s_xb, 62\n.set s_ws, 64\n.set s_xs, 66\n.set s_pws, 68\n.set s_ob, 70\n.set s_ret, 72\n" \
    ".set s_tm0, 74\n.set s_tm1, 76\n" \
      \
    ".macro WORD\n s_mov_b64 exec, 1\n global_store_dword v2, v3, s[26:27] sc0 sc1\n s_mov_b64 exec, -1\n.endm\n" \
      \
    ".macro UNTIL t\n1:\n s_memrealtime s[s_tm0:s_tm0+1]\n s_waitcnt lgkmcnt(0)\n s_sub_u32 s[s_tm0], s[s_tm0], s[\\t]\n s_cmp_lt_i32 s[s_tm0], 0\n" \
    " s_cbranch_scc0 2f\n s_sleep 1\n s_branch 1b\n2:\n.endm\n" \
      \
    ".macro ROWS g\n s_lshl_b32 s[s_rb0], s[\\g], 2\n s_min_u32 s[s_nb], s[\\g], 4\n s_add_u32 s[s_rb0], s[s_rb0], s[s_nb]\n s_cmp_lt_u32 s[\\g], 4\n" \
    " s_cselect_b32 s[s_nb], 5, 4\n.endm\n.macro CALL f\n s_call_b64 s[s_ret:s_ret+1], .La2a_\\f\n.endm\n.macro RET\n s_setpc_b64 s[s_ret:s_ret+1]\n.endm\n" \
      \
    ".macro DW i\n global_load_lds_dwordx4 v[240+\\i], s[s_ws:s_ws+1] offset:\\i*1024\n.endm\n.macro DX j\n" \
    " global_load_lds_dwordx4 v[244+\\j], s[s_xs:s_xs+1] offset:\\j*1024 sc0 sc1\n.endm\n.macro WSRC p\n s_add_u32 m0, s[s_wl], (\\p%%3)*32768\n" \
    " s_lshl_b32 s[s_t0], s[80+\\p], 3\n s_add_u32 s[s_ws], s[s_wb], s[s_t0]\n s_addc_u32 s[s_ws+1], s[s_wb+1], 0\n.endm\n.macro WCONT p\n" \
    " s_add_u32 m0, s[s_wl], (\\p%%3)*32768\n s_nop 0\n.endm\n.macro XSRC p\n s_add_u32 m0, s[s_xl], (\\p%%3)*20480\n s_add_u32 s[s_xs], s[s_xb], s[80+\\p]\n" \
    " s_addc_u32 s[s_xs+1], s[s_xb+1], 0\n.endm\n" \
      \
    ".macro DX2 p\n s_cbranch_vccnz 7f\n DX 2\n s_branch 8f\n7:\n s_mov_b64 exec, (3<<\\p)&0xffff|1\n global_load_dword v238, v11, s[26:27] sc0 sc1\n" \
    " s_mov_b64 exec, -1\n8:\n.endm\n" \
      \
    ".macro RW4 p, d, h\n .irp n, 0,1,2,3\n" \
    " ds_read_b64_tr_b8 v[\\d+8*\\h+2*\\n:\\d+8*\\h+2*\\n+1], v[5+((\\p%%3)&2)+\\h] offset:((\\p%%3)&1)*32768+(\\n>>1)*8192+(\\n&1)*1024\n .endr\n.endm\n" \
    ".macro RX p, b\n .set xr, 88+40*(\\p&1)+8*\\b\n ds_read_b128 v[xr:xr+3], v9 offset:(\\p%%3)*20480+\\b*2048\n" \
    " ds_read_b128 v[xr+4:xr+7], v10 offset:(\\p%%3)*20480+\\b*2048\n.endm\n.macro MM p, w, n\n .set xr, 88+40*(\\p&1)+8*(\\n>>1)\n .if (\\p == 0)\n" \
    " v_mfma_f32_16x16x128_f8f6f4 v[16+4*\\n:19+4*\\n], v[\\w+8*(\\n&1):\\w+7+8*(\\n&1)], v[xr:xr+7], 0\n .else\n" \
    " v_mfma_f32_16x16x128_f8f6f4 v[16+4*\\n:19+4*\\n], v[\\w+8*(\\n&1):\\w+7+8*(\\n&1)], v[xr:xr+7], v[16+4*\\n:19+4*\\n]\n .endif\n.endm\n" \
    ".macro MM5 p, w, n, nx\n s_cmp_eq_u32 s[s_nb], 4\n s_cbranch_scc1 9f\n MM \\p, \\w, \\n\n .if (\\nx >= 0)\n RX \\nx, 4\n .endif\n9:\n.endm\n" \
      \
    ".macro LOOK\n s_cmp_lg_u32 s[s_blind], 0\n s_cbranch_scc1 5f\n s_mov_b64 exec, 0x7c\n global_load_dword v238, v11, s[26:27] sc0 sc1\n" \
    " s_mov_b64 exec, -1\n5:\n.endm\n.macro GATE p\n s_cmp_lg_u32 s[s_blind], 0\n s_cbranch_scc1 4f\n v_readlane_b32 s[s_t0], v238, \\p\n" \
    " s_cmp_ge_u32 s[s_t0], s[s_call]\n s_cbranch_scc1 4f\n v_readlane_b32 s[s_fo], v11, \\p\n CALL spin\n4:\n.endm\n.macro SYNC vm\n" \
    " s_waitcnt vmcnt(\\vm) lgkmcnt(0)\n s_barrier\n.endm\n" \
      \
    ".macro WR n\n ds_write_b128 v[248+(\\n&1)], v[16+4*\\n:19+4*\\n] offset:(\\n>>1)*8192\n.endm\n.macro RD b\n" \
    " ds_read_b128 v[88+8*\\b:91+8*\\b], v250 offset:\\b*8192\n ds_read_b128 v[92+8*\\b:95+8*\\b], v251 offset:\\b*8192\n.endm\n.macro OUT b\n" \
    " .irp e, 0,1,2,3\n v_add_f32 v[88+8*\\b+\\e], v[88+8*\\b+\\e], v[92+8*\\b+\\e]\n .endr\n .irp e, 0,1,2,3\n" \
    " v_mul_f32 v[88+8*\\b+\\e], v253, v[88+8*\\b+\\e]\n .endr\n global_store_dwordx4 v252, v[88+8*\\b:91+8*\\b], s[s_ob:s_ob+1] sc0 sc1\n" \
    " v_add_u32 v252, 0x40000, v252\n.endm\n" \
      \
    " s_load_dwordx16 s[16:31], %[ka], 0\n v_readfirstlane_b32 s[s_wid], %[tid]\n s_mov_b32 s[s_b], %[bid]\n s_and_b32 s[s_t3], %[bid], 7\n" \
    " s_lshl_b32 s[s_t1], s[s_t3], 3\n s_load_dwordx2 s[s_pws:s_pws+1], %[ka], s[s_t1] offset:64\n s_lshr_b32 s[s_wid], s[s_wid], 6\n" \
    " v_and_b32 v4, 63, %[tid]\n v_lshrrev_b32 v3, 4, v4\n v_lshrrev_b32 v2, 3, v4\n s_lshl_b32 s[s_t0], s[s_b], 7\n s_add_u32 s[s_t0], s[s_t0], WS_CNT\n" \
    " v_mov_b32 v12, s[s_t0]\n s_cmp_lt_u32 s[s_b], 32\n s_cbranch_scc0 .La2a_compute\n" \
      \
    " s_lshr_b32 s[s_t0], s[s_b], 3\n s_lshl_b32 s[s_i], s[s_wid], 2\n s_add_u32 s[s_i], s[s_i], s[s_t0]\n s_mul_i32 s[s_q], s[s_i], 37\n" \
    " s_lshr_b32 s[s_q], s[s_q], 8\n s_mul_i32 s[s_t0], s[s_q], 7\n s_sub_u32 s[s_i], s[s_i], s[s_t0]\n s_sub_u32 s[s_t2], s[s_t3], 1\n ROWS s_t2\n" \
    " s_mov_b32 s[s_t1], s[s_rb0]\n s_add_u32 s[s_g], s[s_i], s[s_t2]\n s_sub_u32 s[s_t0], s[s_g], 7\n s_min_u32 s[s_g], s[s_g], s[s_t0]\n ROWS s_g\n" \
    " s_sub_u32 s[s_t0], s[s_rb0], s[s_t1]\n s_and_b32 s[s_t0], s[s_t0], 31\n s_lshl_b32 s[s_t1], s[s_q], 5\n s_add_u32 s[s_t0], s[s_t0], s[s_t1]\n" \
    " s_mul_i32 s[s_t0], s[s_t0], " STR(SLOT_DT) "*205\n s_lshr_b32 s[s_r], s[s_t0], 11\n s_add_u32 s[s_t1], s[s_t1], s[s_rb0]\n" \
    " s_lshl_b32 s[s_t1], s[s_t1], 11\n v_lshl_add_u32 v11, v4, 4, s[s_t1]\n v_and_b32 v10, 7, v4\n v_lshlrev_b32 v10, 4, v10\n" \
    " v_lshl_add_u32 v10, v2, 9, v10\n s_lshl_b32 s[s_t1], s[s_rb0], 13\n s_lshl_b32 s[s_t2], s[s_q], 7\n s_add_u32 s[s_t1], s[s_t1], s[s_t2]\n" \
    " v_add_u32 v10, s[s_t1], v10\n s_lshl_b32 s[s_t1], s[s_g], 9\n s_add_u32 s[s_t2], s[s_t2], s[s_t1]\n s_waitcnt lgkmcnt(0)\n" \
    " s_add_u32 s[s_t1], s28, s[s_t3]\n s_and_b32 s[s_t1], s[s_t1], 7\n s_lshl_b32 s[s_t1], s[s_t1], 18\n s_add_u32 s[s_xs], s16, s[s_t1]\n" \
    " s_addc_u32 s[s_xs+1], s17, 0\n s_cmp_lt_u32 s[s_wid], 2\n s_cbranch_scc0 4f\n CALL pld\n4:\n s_lshl_b32 s[s_t0], s28, 18\n" \
    " s_add_u32 s[s_t0], s[s_t0], WS_X\n s_lshl_b32 s[s_t1], s28, 12\n s_add_u32 s[s_t1], s[s_t1], WS_XF\n" \
      \
    " s_cmp_eq_u32 s[s_wid], 7\n s_cbranch_scc0 4f\n v_lshl_add_u32 v5, v4, 21, s[s_t0]\n s_add_u32 s[s_t2], s[s_t1], 3584\n v_writelane_b32 v5, s[s_t2], 8\n" \
    " s_mov_b64 exec, 0x1ff\n global_load_dword v6, v5, s[s_pws:s_pws+1] sc0 sc1\n s_mov_b64 exec, -1\n4:\n global_load_dword v3, v12, s[26:27] sc1\n" \
    " s_add_u32 s[s_t1], s[s_t1], s[s_t2]\n v_mov_b32 v7, s[s_t1]\n s_add_u32 s[s_ob], s[s_pws], s[s_t0]\n s_addc_u32 s[s_ob+1], s[s_pws+1], 0\n" \
    " s_waitcnt vmcnt(0)\n s_barrier\n s_memrealtime s[s_tm1:s_tm1+1]\n v_add_u32 v3, 1, v3\n s_cmp_eq_u32 s[s_wid], 7\n s_cbranch_scc0 4f\n" \
    " v_mov_b32 v2, v12\n WORD\n s_branch .La2a_pend\n4:\n v_and_b32 v12, 7, v3\n v_lshl_add_u32 v11, v12, 21, v11\n v_add_u32 v12, 4096, v11\n" \
    " v_add_u32 v13, 8192, v11\n s_cmp_eq_u32 s[s_t3], 0\n s_cbranch_scc1 .La2a_pend\n s_waitcnt lgkmcnt(0)\n s_add_u32 s[s_r], s[s_r], s[s_tm1]\n" \
    " s_cmp_lt_u32 s[s_wid], 2\n s_cbranch_scc1 4f\n s_sub_u32 s[s_t1], s[s_r], " STR(LOAD_LEAD) "\n UNTIL s_t1\n CALL pld\n s_waitcnt vmcnt(0)\n4:\n" \
    " UNTIL s_r\n" \
      \
    " .irp j, 0,1,2,3,4,5,6,7,8,9\n .if (\\j == 8)\n s_cmp_eq_u32 s[s_nb], 4\n s_cbranch_scc1 3f\n .endif\n" \
    " global_store_dwordx4 v[11+(\\j>>2)], v[24+4*\\j:27+4*\\j], s[s_ob:s_ob+1] offset:(\\j&3)*1024 sc0 sc1\n .endr\n3:\n s_waitcnt vmcnt(0)\n" \
    " s_mov_b64 exec, 1\n global_store_dword v7, v3, s[s_pws:s_pws+1] sc0 sc1\n.La2a_pend:\n s_endpgm\n.La2a_pld:\n s_cmp_eq_u32 s[s_t3], 0\n" \
    " s_cbranch_scc1 3f\n .irp j, 0,1,2,3,4,5,6,7,8,9\n .if (\\j == 8)\n s_cmp_eq_u32 s[s_nb], 4\n s_cbranch_scc1 3f\n .endif\n v_add_u32 v5, \\j*4096, v10\n" \
    " global_load_dwordx4 v[24+4*\\j:27+4*\\j], v5, s[s_xs:s_xs+1]\n .endr\n3:\n RET\n" \
      \
    ".La2a_compute:\n s_lshr_b32 s[s_g], s[s_b], 5\n s_sub_u32 s[s_g], s[s_g], 1\n s_bfe_u32 s[s_q], s[s_b], 0x20003\n s_lshl_b32 s[s_nt], s[s_t3], 2\n" \
    " s_add_u32 s[s_nt], s[s_nt], s[s_q]\n s_and_b32 s[s_wn], s[s_wid], 3\n s_lshr_b32 s[s_wk], s[s_wid], 2\n ROWS s_g\n v_and_b32 v1, 7, v4\n" \
    " v_xor_b32 v1, v1, v3\n v_lshlrev_b32 v1, 4, v1\n v_lshl_add_u32 v240, v2, 12, v1\n v_xor_b32 v5, 64, v240\n v_add_u32 v241, 31744, v240\n" \
    " v_add_u32 v242, 2*31744, v5\n v_add_u32 v243, 3*31744, v5\n s_lshl_b32 s[s_wl], s[s_wid], 12\n s_lshl_b32 s[s_t0], s[s_nb], 1\n" \
    " s_sub_u32 s[s_t0], s[s_t0], 8\n s_cmp_ge_u32 s[s_wn], s[s_t0]\n s_cselect_b64 vcc, -1, 0\n s_min_u32 s[s_t0], s[s_t0], s[s_wn]\n" \
    " s_lshl_b32 s[s_xp], s[s_wn], 1\n s_add_u32 s[s_xp], s[s_xp], s[s_t0]\n s_and_b32 s[s_t0], s[s_xp], 1\n s_lshl_b32 s[s_t0], s[s_t0], 6\n" \
    " v_xor_b32 v247, s[s_t0], v1\n v_lshl_add_u32 v244, v2, 9, v247\n v_xor_b32 v245, 64, v244\n v_add_u32 v245, 3072, v245\n v_add_u32 v246, 2*3072, v244\n" \
    " v_lshl_add_u32 v247, v2, 7, v247\n s_mul_i32 s[s_t2], s[s_wk], 10240\n s_add_u32 s[s_t2], s[s_t2], L_X\n s_lshl_b32 s[s_xl], s[s_xp], 10\n" \
    " s_add_u32 s[s_xl], s[s_xl], s[s_t2]\n s_lshl_b32 s[s_t0], s[s_rb0], 1\n s_add_u32 s[s_xp], s[s_xp], s[s_t0]\n" \
      \
    " v_and_b32 v7, 15, v4\n v_lshrrev_b32 v8, 1, v7\n v_lshl_add_u32 v9, v3, 4, v8\n v_lshlrev_b32 v9, 7, v9\n v_lshrrev_b32 v10, 1, v8\n" \
    " v_and_b32 v5, 1, v3\n v_lshl_add_u32 v10, v5, 2, v10\n s_lshl_b32 s[s_t0], s[s_wn], 1\n v_xor_b32 v10, s[s_t0], v10\n v_lshl_add_u32 v9, v10, 4, v9\n" \
    " v_and_b32 v10, 1, v4\n v_lshl_add_u32 v9, v10, 3, v9\n s_lshl_b32 s[s_t0], s[s_wk], 14\n v_add_u32 v5, s[s_t0], v9\n v_xor_b32 v8, v8, v3\n" \
    " v_lshlrev_b32 v8, 4, v8\n v_lshl_add_u32 v8, v7, 7, v8\n v_add_u32 v9, s[s_t2], v8\n v_xor_b32 v10, 64, v9\n v_lshl_add_u32 v248, v7, 9, v1\n" \
    " v_xor_b32 v6, 16, v5\n v_add_u32 v7, 65536, v5\n v_add_u32 v8, 65536, v6\n s_mul_i32 s[s_t0], s[s_wk], 40960\n s_lshl_b32 s[s_t1], s[s_wn], 7\n" \
    " s_add_u32 s[s_t0], s[s_t0], s[s_t1]\n v_add_u32 v248, s[s_t0], v248\n v_xor_b32 v249, 64, v248\n v_lshrrev_b32 v252, 5, v4\n" \
    " v_lshl_add_u32 v252, s[s_wid], 1, v252\n v_and_b32 v253, 31, v4\n v_and_b32 v250, 7, v252\n v_xor_b32 v250, v250, v253\n v_lshlrev_b32 v250, 4, v250\n" \
    " v_lshl_add_u32 v250, v252, 9, v250\n v_add_u32 v251, 40960, v250\n v_lshlrev_b32 v253, 4, v253\n v_lshl_add_u32 v252, v252, 14, v253\n" \
      \
    " s_sub_u32 s[s_t0], s[s_wk], 4\n v_lshl_add_u32 v13, v4, 1, s[s_t0]\n v_and_b32 v13, 63, v13\n v_mul_u32_u24 v236, 37, v13\n" \
    " v_lshrrev_b32 v236, 8, v236\n v_mul_u32_u24 v237, 7, v236\n v_sub_u32 v237, v13, v237\n v_sub_u32 v237, s[s_g], v237\n v_add_u32 v13, 7, v237\n" \
    " v_min_u32 v237, v237, v13\n v_mov_b32 v255, WS_TMO\n v_mov_b32 v11, v255\n v_mov_b32 v238, 0\n s_cmp_eq_u32 s[s_wn], 3\n" \
    " s_cselect_b32 s[s_blind], 0, -1\n s_add_u32 s[s_q], s[s_q], s[s_g]\n s_and_b32 s[s_q], s[s_q], 3\n s_xor_b32 s[s_q], s[s_q], s[s_wk]\n" \
    " s_lshl_b32 s[s_i], s[s_q], 19\n s_lshl_b32 s[s_r], s[s_xp], 12\n s_lshl_b32 s[s_t1], s[s_q], 7\n s_add_u32 s[s_r], s[s_r], s[s_t1]\n" \
    " s_lshl_b32 s[s_t0], s[s_wn], 17\n s_lshl_b32 s[s_t1], s[s_nt], 7\n s_add_u32 s[s_t2], s[s_t0], s[s_t1]\n s_mov_b32 m0, s[s_wl]\n s_waitcnt lgkmcnt(0)\n" \
      \
    " s_sleep 8\n s_add_u32 s[s_wb], s18, s[s_t2]\n s_addc_u32 s[s_wb+1], s19, 0\n s_add_u32 s[s_i], s[s_i], s29\n s_add_u32 s[s_ws], s[s_wb], s[s_i]\n" \
    " s_addc_u32 s[s_ws+1], s[s_wb+1], 0\n DW 0\n DW 1\n DW 2\n DW 3\n s_mov_b32 m0, s[s_xl]\n s_add_u32 s[s_r], s[s_r], s30\n" \
    " s_add_u32 s[s_xs], s16, s[s_r]\n s_addc_u32 s[s_xs+1], s17, 0\n DX 0\n DX 1\n s_cbranch_vccnz 3f\n DX 2\n3:\n" \
    " global_load_dword v12, v12, s[26:27] sc1\n s_load_dword s32, s[20:21], 0\n s_load_dword s33, s[22:23], 0\n s_add_u32 m0, s[s_xl], 20480\n" \
    " s_xor_b32 s[s_r], s[s_r], 256\n s_add_u32 s[s_xs], s16, s[s_r]\n s_addc_u32 s[s_xs+1], s17, 0\n s_xor_b32 s[s_i], s[s_i], 2<<19\n" \
    " s_add_u32 s[s_ws], s[s_wb], s[s_i]\n s_addc_u32 s[s_ws+1], s[s_wb+1], 0\n s_barrier\n DX 0\n DX 1\n s_cbranch_vccnz 3f\n DX 2\n3:\n" \
    " s_add_u32 m0, s[s_wl], 32768\n v_sub_u32 v237, s31, v237\n DW 0\n v_and_b32 v237, 7, v237\n v_lshl_add_u32 v13, v237, 2, v236\n" \
    " v_lshlrev_b32 v237, 12, v237\n v_lshl_add_u32 v237, v236, 7, v237\n DW 1\n s_lshl_b32 s[s_t0], s[s_g], 9\n s_add_u32 s[s_t0], s[s_t0], WS_XF\n" \
    " s_mov_b32 exec_hi, 0\n s_andn2_b32 exec_lo, 0xfffc, s[s_blind]\n v_add_u32 v11, s[s_t0], v237\n s_mov_b64 exec, -1\n DW 2\n" \
    " v_lshlrev_b32 v236, 16, v13\n v_mov_b32 v244, v247\n .irp p, 2,3,4,5,6,7,8,9,10,11,12,13,14,15\n v_readlane_b32 s[80+\\p], v236, \\p\n .endr\n" \
    " v_xor_b32 v245, 64, v247\n v_mov_b32 v246, v247\n s_lshl_b32 s[s_t0], s[s_rb0], 18\n s_lshl_b32 s[s_t1], s[s_nt], 9\n" \
    " s_add_u32 s[s_t0], s[s_t0], s[s_t1]\n s_add_u32 s[s_ob], s24, s[s_t0]\n s_addc_u32 s[s_ob+1], s25, 0\n DW 3\n" \
      \
    " WSRC 2\n SYNC 6\n RW4 0, 56, 0\n RX 0, 0\n RW4 0, 56, 1\n RX 0, 1\n RX 0, 2\n RX 0, 3\n RX 0, 4\n DW 0\n DW 1\n DW 2\n DW 3\n" \
    " v_readfirstlane_b32 s[s_call], v12\n s_add_u32 s[s_call], s[s_call], 1\n s_and_b32 s[s_t0], s[s_call], 7\n s_lshl_b32 s[s_t0], s[s_t0], 21\n" \
    " s_lshl_b32 s[s_t1], s[s_xp], 10\n s_add_u32 s[s_t0], s[s_t0], s[s_t1]\n s_add_u32 s[s_t0], s[s_t0], WS_X\n s_add_u32 s[s_xb], s26, s[s_t0]\n" \
    " s_addc_u32 s[s_xb+1], s27, 0\n v_mov_b32 v253, s32\n v_mul_f32 v253, s33, v253\n" \
    " WSRC 3\n SYNC 4\n RW4 1, 72, 0\n RX 1, 0\n RW4 1, 72, 1\n RX 1, 1\n RX 1, 2\n RX 1, 3\n RX 1, 4\n DW 0\n MM 0, 56, 0\n DW 1\n MM 0, 56, 1\n DW 2\n" \
    " MM 0, 56, 2\n DW 3\n MM 0, 56, 3\n" \
    " WSRC 4\n SYNC 4\n RW4 2, 168, 0\n RW4 2, 168, 1\n DW 0\n MM 0, 56, 4\n DW 1\n MM 0, 56, 5\n DW 2\n MM 0, 56, 6\n DW 3\n MM 0, 56, 7\n" \
    " WSRC 5\n SYNC 4\n RW4 3, 184, 0\n RW4 3, 184, 1\n DW 0\n MM5 0, 56, 8, -1\n DW 1\n MM5 0, 56, 9, -1\n DW 2\n MM 1, 72, 0\n DW 3\n MM 1, 72, 1\n" \
    " WSRC 6\n SYNC 4\n RW4 4, 200, 0\n RW4 4, 200, 1\n DW 0\n MM 1, 72, 2\n DW 1\n MM 1, 72, 3\n DW 2\n MM 1, 72, 4\n DW 3\n MM 1, 72, 5\n" \
    " WSRC 7\n SYNC 4\n RW4 5, 216, 0\n RW4 5, 216, 1\n DW 0\n MM 1, 72, 6\n DW 1\n MM 1, 72, 7\n DW 2\n MM5 1, 72, 8, -1\n DW 3\n MM5 1, 72, 9, -1\n" \
    " WSRC 8\n SYNC 4\n LOOK\n RW4 6, 56, 0\n RW4 6, 56, 1\n DW 0\n DW 1\n DW 2\n DW 3\n" \
    " WSRC 9\n SYNC 4\n LOOK\n RW4 7, 72, 0\n RW4 7, 72, 1\n DW 0\n DW 1\n DW 2\n DW 3\n" \
    " XSRC 2\n GATE 2\n SYNC 4\n DX2 3\n DX 0\n DX 1\n WSRC 10\n DW 0\n DW 1\n DW 2\n DW 3\n" \
    " XSRC 3\n GATE 3\n SYNC 7\n DX2 4\n DX 0\n DX 1\n" \
    " XSRC 4\n GATE 4\n SYNC 3\n RX 2, 0\n RX 2, 1\n RX 2, 2\n RX 2, 3\n RX 2, 4\n DX2 5\n DX 0\n DX 1\n" \
    " XSRC 5\n GATE 5\n SYNC 3\n MM 2, 168, 0\n RX 3, 0\n DX2 6\n MM5 2, 168, 8, 3\n MM 2, 168, 1\n RX 3, 1\n MM 2, 168, 2\n RX 3, 2\n DX 0\n MM 2, 168, 3\n" \
    " RX 3, 3\n MM5 2, 168, 9, -1\n MM 2, 168, 4\n DX 1\n MM 2, 168, 5\n MM 2, 168, 6\n GATE 6\n XSRC 6\n MM 2, 168, 7\n" \
    " SYNC 3\n MM 3, 184, 0\n RW4 8, 168, 0\n DX2 7\n MM5 3, 184, 8, 4\n MM 3, 184, 1\n RX 4, 0\n MM 3, 184, 2\n RW4 8, 168, 1\n DX 0\n MM 3, 184, 3\n" \
    " RX 4, 1\n MM5 3, 184, 9, -1\n MM 3, 184, 4\n RX 4, 2\n DX 1\n MM 3, 184, 5\n RX 4, 3\n MM 3, 184, 6\n GATE 7\n XSRC 7\n MM 3, 184, 7\n" \
    " SYNC 3\n MM 4, 200, 0\n RW4 9, 184, 0\n DX2 8\n MM5 4, 200, 8, 5\n MM 4, 200, 1\n RX 5, 0\n DX 0\n MM 4, 200, 2\n RW4 9, 184, 1\n DX 1\n MM 4, 200, 3\n" \
    " RX 5, 1\n MM5 4, 200, 9, -1\n WSRC 11\n DW 0\n MM 4, 200, 4\n RX 5, 2\n MM 4, 200, 5\n RX 5, 3\n DW 1\n MM 4, 200, 6\n DW 2\n GATE 8\n XSRC 8\n" \
    " MM 4, 200, 7\n" \
    " SYNC 6\n MM 5, 216, 0\n RW4 10, 200, 0\n DX2 9\n MM5 5, 216, 8, 6\n MM 5, 216, 1\n RX 6, 0\n DX 0\n MM 5, 216, 2\n RW4 10, 200, 1\n DX 1\n" \
    " MM 5, 216, 3\n RX 6, 1\n MM5 5, 216, 9, -1\n WCONT 11\n DW 3\n MM 5, 216, 4\n RX 6, 2\n MM 5, 216, 5\n RX 6, 3\n WSRC 12\n DW 0\n MM 5, 216, 6\n DW 1\n" \
    " GATE 9\n XSRC 9\n MM 5, 216, 7\n" \
    " SYNC 6\n MM 6, 56, 0\n RX 7, 0\n DX2 10\n MM5 6, 56, 8, 7\n MM 6, 56, 1\n RX 7, 1\n DX 0\n MM 6, 56, 2\n RX 7, 2\n DX 1\n MM 6, 56, 3\n RX 7, 3\n" \
    " MM5 6, 56, 9, -1\n WCONT 12\n DW 2\n MM 6, 56, 4\n MM 6, 56, 5\n DW 3\n MM 6, 56, 6\n WSRC 13\n DW 0\n GATE 10\n XSRC 10\n MM 6, 56, 7\n" \
    " SYNC 6\n MM 7, 72, 0\n RW4 11, 216, 0\n DX2 11\n MM5 7, 72, 8, 8\n MM 7, 72, 1\n RX 8, 0\n DX 0\n MM 7, 72, 2\n RW4 11, 216, 1\n DX 1\n MM 7, 72, 3\n" \
    " RX 8, 1\n MM5 7, 72, 9, -1\n WCONT 13\n DW 1\n MM 7, 72, 4\n RX 8, 2\n MM 7, 72, 5\n RX 8, 3\n DW 2\n MM 7, 72, 6\n DW 3\n GATE 11\n XSRC 11\n" \
    " MM 7, 72, 7\n" \
    " SYNC 6\n MM 8, 168, 0\n RW4 12, 56, 0\n DX2 12\n MM5 8, 168, 8, 9\n MM 8, 168, 1\n RX 9, 0\n DX 0\n MM 8, 168, 2\n RW4 12, 56, 1\n MM 8, 168, 3\n" \
    " RX 9, 1\n DX 1\n MM5 8, 168, 9, -1\n MM 8, 168, 4\n RX 9, 2\n WSRC 14\n DW 0\n MM 8, 168, 5\n RX 9, 3\n MM 8, 168, 6\n DW 1\n GATE 12\n XSRC 12\n" \
    " MM 8, 168, 7\n" \
    " SYNC 5\n MM 9, 184, 0\n RW4 13, 72, 0\n DX2 13\n MM5 9, 184, 8, 10\n MM 9, 184, 1\n RX 10, 0\n DX 0\n MM 9, 184, 2\n RW4 13, 72, 1\n MM 9, 184, 3\n" \
    " RX 10, 1\n DX 1\n MM5 9, 184, 9, -1\n MM 9, 184, 4\n RX 10, 2\n WCONT 14\n DW 2\n MM 9, 184, 5\n RX 10, 3\n MM 9, 184, 6\n DW 3\n GATE 13\n XSRC 13\n" \
    " MM 9, 184, 7\n" \
    " SYNC 5\n MM 10, 200, 0\n RX 11, 0\n DX2 14\n MM5 10, 200, 8, 11\n MM 10, 200, 1\n RX 11, 1\n DX 0\n MM 10, 200, 2\n RX 11, 2\n MM 10, 200, 3\n" \
    " RX 11, 3\n DX 1\n MM5 10, 200, 9, -1\n MM 10, 200, 4\n WSRC 15\n DW 0\n MM 10, 200, 5\n MM 10, 200, 6\n DW 1\n GATE 14\n XSRC 14\n MM 10, 200, 7\n" \
    " SYNC 5\n MM 11, 216, 0\n RW4 14, 168, 0\n DX2 15\n MM5 11, 216, 8, 12\n MM 11, 216, 1\n RX 12, 0\n DX 0\n MM 11, 216, 2\n RW4 14, 168, 1\n" \
    " MM 11, 216, 3\n RX 12, 1\n DX 1\n MM5 11, 216, 9, -1\n MM 11, 216, 4\n RX 12, 2\n WCONT 15\n DW 2\n MM 11, 216, 5\n RX 12, 3\n MM 11, 216, 6\n DW 3\n" \
    " GATE 15\n XSRC 15\n MM 11, 216, 7\n" \
    " SYNC 5\n MM 12, 56, 0\n RX 13, 0\n DX2 16\n MM5 12, 56, 8, 13\n MM 12, 56, 1\n RX 13, 1\n MM 12, 56, 2\n RX 13, 2\n DX 0\n MM 12, 56, 3\n RX 13, 3\n" \
    " MM5 12, 56, 9, -1\n MM 12, 56, 4\n DX 1\n MM 12, 56, 5\n MM 12, 56, 6\n MM 12, 56, 7\n" \
    " SYNC 3\n MM 13, 72, 0\n RW4 15, 184, 0\n MM5 13, 72, 8, 14\n MM 13, 72, 1\n RX 14, 0\n MM 13, 72, 2\n RW4 15, 184, 1\n MM 13, 72, 3\n RX 14, 1\n" \
    " MM5 13, 72, 9, -1\n MM 13, 72, 4\n RX 14, 2\n MM 13, 72, 5\n RX 14, 3\n MM 13, 72, 6\n MM 13, 72, 7\n" \
      \
    " SYNC 0\n global_load_dword v12, v255, s[26:27] sc0 sc1\n MM 14, 168, 0\n RX 15, 0\n RX 15, 1\n MM 14, 168, 1\n RX 15, 2\n MM 14, 168, 2\n RX 15, 3\n" \
    " MM 14, 168, 3\n RX 15, 4\n s_waitcnt lgkmcnt(6)\n MM 15, 184, 0\n MM 15, 184, 1\n MM 15, 184, 2\n MM 15, 184, 3\n s_nop 8\n WR 0\n MM 14, 168, 4\n" \
    " WR 1\n MM 14, 168, 5\n WR 2\n s_waitcnt lgkmcnt(3)\n MM 15, 184, 4\n WR 3\n MM 15, 184, 5\n s_waitcnt vmcnt(0)\n v_cmp_eq_u32 vcc, s[s_call], v12\n" \
    " s_cmp_lg_u32 vcc_hi, 0\n s_cbranch_scc0 3f\n v_mov_b32 v253, -1\n3:\n MM 14, 168, 6\n s_waitcnt lgkmcnt(0)\n s_barrier\n MM 14, 168, 7\n RD 0\n RD 1\n" \
    " WR 4\n MM 15, 184, 6\n WR 5\n MM 15, 184, 7\n s_waitcnt lgkmcnt(4)\n OUT 0\n s_cmp_eq_u32 s[s_nb], 4\n s_cbranch_scc1 .La2a_nb4\n MM 14, 168, 8\n" \
    " WR 6\n s_waitcnt lgkmcnt(3)\n OUT 1\n MM 14, 168, 9\n WR 7\n MM 15, 184, 8\n MM 15, 184, 9\n s_waitcnt lgkmcnt(0)\n s_barrier\n RD 2\n RD 3\n s_nop 5\n" \
    " WR 8\n WR 9\n s_waitcnt lgkmcnt(4)\n OUT 2\n s_waitcnt lgkmcnt(0)\n s_barrier\n RD 4\n OUT 3\n s_waitcnt lgkmcnt(0)\n OUT 4\n s_branch 3f\n.La2a_nb4:\n" \
    " WR 6\n WR 7\n s_waitcnt lgkmcnt(4)\n OUT 1\n s_waitcnt lgkmcnt(0)\n s_barrier\n RD 2\n RD 3\n s_waitcnt lgkmcnt(2)\n OUT 2\n s_waitcnt lgkmcnt(0)\n" \
    " OUT 3\n3:\n s_cmp_eq_u32 s[s_wid], 0\n s_cbranch_scc0 3f\n s_lshl_b32 s[s_t0], s[s_b], 7\n s_add_u32 s[s_t0], s[s_t0], WS_CNT\n v_mov_b32 v2, s[s_t0]\n" \
    " v_mov_b32 v3, s[s_call]\n WORD\n3:\n s_endpgm\n" \
      \
    ".La2a_spin:\n s_memrealtime s[s_tm0:s_tm0+1]\n v_writelane_b32 v255, s[s_fo], 0\n s_cmp_eq_u32 s[s_q], -1\n s_cbranch_scc1 .La2a_spin1\n" \
    " s_mov_b32 s[s_q], -1\n s_sub_u32 s[s_t1], 13, s[s_wk]\n v_readlane_b32 s[s_t0], v13, s[s_t1]\n s_lshl_b32 s[s_t1], s[s_wk], 18\n" \
    " s_lshl_b32 s[s_t0], s[s_t0], 19\n s_add_u32 s[s_t0], s[s_t0], s[s_t1]\n s_sub_u32 s[s_t0], s[s_t0], 3<<17\n s_add_u32 s[s_tm1], s[s_wb], s[s_t0]\n" \
    " s_addc_u32 s[s_tm1+1], s[s_wb+1], 0\n .rept 2\n .irp i, 0,1,2,3\n global_load_dwordx4 v[232:235], v[240+\\i], s[s_tm1:s_tm1+1] offset:\\i*1024\n" \
    " .endr\n s_add_u32 s[s_tm1], s[s_tm1], 1<<17\n s_addc_u32 s[s_tm1+1], s[s_tm1+1], 0\n .endr\n.La2a_spin1:\n" \
    " global_load_dword v12, v255, s[26:27] sc0 sc1\n LOOK\n s_memrealtime s[s_tm1:s_tm1+1]\n s_waitcnt vmcnt(0) lgkmcnt(0)\n" \
    " s_sub_u32 s[s_t1], s[s_tm1], s[s_tm0]\n v_readlane_b32 s[s_pk], v12, 0\n v_readlane_b32 s[s_t0], v12, 1\n s_cmp_ge_u32 s[s_pk], s[s_call]\n" \
    " s_cbranch_scc1 .La2a_spin3\n s_cmp_eq_u32 s[s_t0], s[s_call]\n s_cbranch_scc1 .La2a_spin3\n s_cmp_lt_u32 s[s_t1], " STR(DEADLINE_TICKS) "\n s_sleep 2\n" \
    " s_cbranch_scc1 .La2a_spin1\n v_mov_b32 v2, WS_TMO\n v_mov_b32 v3, s[s_call]\n WORD\n s_waitcnt vmcnt(0)\n.La2a_spin3:\n RET\n"

__global__ void __launch_bounds__(NTHREADS, 2) __attribute__((amdgpu_num_vgpr(2))) a2a_gemm(Params p) {
    (void)p;
    asm volatile(BODY_ASM
        :: [ka] "s"(__builtin_amdgcn_kernarg_segment_ptr()), [bid] "s"(blockIdx.x), [tid] "v"(threadIdx.x)
        : "memory", "vcc", "scc", "v255",
          "s16", "s17", "s18", "s19", "s20", "s21", "s22", "s23", "s24", "s25", "s26", "s27", "s28", "s29", "s30", "s31",
          "s32", "s33", "s34", "s35", "s36", "s37", "s38", "s39", "s40", "s41", "s42", "s43", "s44", "s45", "s46", "s47",
          "s48", "s49", "s50", "s51", "s52", "s53", "s54", "s55", "s56", "s57", "s58", "s59", "s60", "s61", "s62", "s63",
          "s64", "s65", "s66", "s67", "s68", "s69", "s70", "s71", "s72", "s73", "s74", "s75", "s76", "s77", "s78", "s79",
          "s80", "s81", "s82", "s83", "s84", "s85", "s86", "s87", "s88", "s89", "s90", "s91", "s92", "s93", "s94", "s95");
}

extern "C" void kernel_launch(int rank, int nranks, ncclComm_t comm,
                              hipStream_t stream,
                              void* const* d_in, const int* in_sizes, int n_in,
                              void* d_out, int out_size,
                              const int* in_dtypes, int out_dtype,
                              void* const* d_out_all,
                              void* const* d_ws_all, size_t ws_bytes) {
    (void)comm; (void)in_dtypes; (void)out_dtype; (void)d_out_all;
    if (nranks != NR || (unsigned)rank >= (unsigned)NR || n_in != 4 || ws_bytes < (size_t)WS_END ||
        in_sizes[0] != 4096 * KS || in_sizes[1] != 4096 * GN || out_size != ML * GN) {
        fprintf(stderr, "unexpected problem shape\n");
        return;
    }
    HIP_CHECK(hipFuncSetAttribute((const void*)a2a_gemm, hipFuncAttributeMaxDynamicSharedMemorySize, L_END));
    Params p;
    p.x = d_in[0];
    p.w = d_in[1];
    p.sx = d_in[2];
    p.sw = d_in[3];
    p.out = d_out;
    p.myws = d_ws_all[rank];
    p.rank = (unsigned)rank;
    p.w_own = (unsigned)rank * KS * GN;
    p.x_own = (unsigned)rank * ML * KS;
    p.rank_m1 = (unsigned)rank + NR - 1;
    for (int i = 0; i < NR; ++i) p.ws[i] = d_ws_all[(rank + i) % NR];
    a2a_gemm<<<NWG, NTHREADS, L_END, stream>>>(p);
    HIP_CHECK(hipGetLastError());
}
```
